```python
import numpy as np

kernel_name = "gpu_gemm_rs_m2048_k2048_n2048_bf16_relu"


ABSMAX_RTOL = 0.008

COMPUTE_DTYPE = "bfloat16"
EPILOGUE = "relu"
IN_QUANT = None
OUT_QUANT = None
IN_DTYPES = {"x": "bfloat16", "w_mat": "bfloat16"}
WORLD_SIZE = 8

def _to_bf16(x):
    u = np.ascontiguousarray(x, dtype=np.float32).view(np.uint32)
    return ((u + 0x7FFF + ((u >> 16) & 1)) & 0xFFFF0000).view(np.float32)

def setup_inputs():

    x = _to_bf16(np.random.randn(2048, 2048).astype(np.float32))
    w_mat = _to_bf16((np.random.randn(2048, 2048) / np.sqrt(2048)).astype(np.float32))
    return {"x": x, "w_mat": w_mat}

def _epilogue(y):
    return np.maximum(y, 0.0).astype(np.float32)

def reference(x, w_mat):
    y = (x @ w_mat).astype(np.float32)
    return _epilogue(y)
```

```cpp
#include <hip/hip_runtime.h>
#include <rccl/rccl.h>
#include <cstdio>

typedef unsigned u32;
typedef unsigned long long u64;
typedef u32 u32x4 __attribute__((ext_vector_type(4)));
typedef u32 u32x2 __attribute__((ext_vector_type(2)));
typedef short s16x4 __attribute__((ext_vector_type(4)));
typedef __bf16 bf16x8 __attribute__((ext_vector_type(8)));
typedef __bf16 bf16x2 __attribute__((ext_vector_type(2)));
typedef float f32x2 __attribute__((ext_vector_type(2)));
typedef float f32x16 __attribute__((ext_vector_type(16)));
typedef float f32x4 __attribute__((ext_vector_type(4)));
typedef __amdgpu_buffer_rsrc_t rsrc_t;
#define LDSP(T, p) ((__attribute__((address_space(3))) T*)(p))

constexpr int NR = 8;
constexpr u32 KIB = 1024u, MIB = 1024u * 1024u;
constexpr u32 WS_CTR = 0;
constexpr u32 WS_TMO = 64 * KIB;
constexpr u32 WS_LOG = 128 * KIB;
constexpr u32 WS_OB = 1 * MIB;
constexpr u32 WS_IB = 3 * MIB;
constexpr u32 IB_BYTES = 4 * MIB;
constexpr u32 WS_NEED = WS_IB + 3 * IB_BYTES;
#define NPUSH 32
#define NGEMM 192
constexpr int N_PUSH = NPUSH, N_GEMM = NGEMM, N_WG = 256;
constexpr int N_RED = N_WG - N_PUSH - N_GEMM, RED0 = N_PUSH + N_GEMM;
constexpr u32 N_PAIR = 32 - N_GEMM / 8;
constexpr u32 LDS_BYTES = 128 * KIB;
#define DEFER 0
#define RED_PAR 2
#define OPT_G 4u
#define GEMM_SLEEP 2
#define RED_SLEEP 4
constexpr u64 DEADLINE_TICKS = 20000000ull;
constexpr int AUX_SYS = 17;

struct Params {
    char* ws[NR];
    const char* x;
    const char* w;
    char* out;
    int rank;
};

__device__ __forceinline__ rsrc_t mk_rsrc(const void* p, u32 bytes) {
    return __builtin_amdgcn_make_buffer_rsrc((void*)p, 0, bytes, 0x00020000);
}
__device__ __forceinline__ void st16(u32x4 v, rsrc_t r, u32 voff, u32 soff) {
    __builtin_amdgcn_raw_buffer_store_b128(v, r, voff + soff, 0, AUX_SYS);
}
__device__ __forceinline__ u32 nz(u32 d) { return d ? d : 0x80000000u; }
__device__ __forceinline__ u32x4 nz4(u32x4 v) { return u32x4{nz(v.x), nz(v.y), nz(v.z), nz(v.w)}; }
__device__ __forceinline__ u32 min4(u32 m, u32x4 v) { return min(min(m, min(v.x, v.y)), min(v.z, v.w)); }
__device__ __forceinline__ u32 pk_bf16(float a, float b) {
    return __builtin_bit_cast(u32, __builtin_convertvector(f32x2{a, b}, bf16x2));
}
__device__ __forceinline__ float bf_lo(u32 d) { return __builtin_bit_cast(float, d << 16); }
__device__ __forceinline__ float bf_hi(u32 d) { return __builtin_bit_cast(float, d & 0xffff0000u); }
__device__ __forceinline__ float relu(float v) { return v < 0.0f ? 0.0f : v; }
__device__ __forceinline__ u32 add_relu(u32 a, u32 b) {
    return pk_bf16(relu(bf_lo(a) + bf_lo(b)), relu(bf_hi(a) + bf_hi(b)));
}

struct Waiter {
    u32* tmo; u32 call; u64 t0; u32 spins;
    __device__ __forceinline__ bool expired() {
        asm volatile("" ::: "memory");
        if ((++spins & 63u) != 0) return false;
        u32 t = __hip_atomic_load(tmo, __ATOMIC_RELAXED, __HIP_MEMORY_SCOPE_AGENT);
        t = __builtin_amdgcn_readfirstlane(t);
        if (t == call || __builtin_amdgcn_s_memrealtime() - t0 > DEADLINE_TICKS) {
            __hip_atomic_store(tmo, call, __ATOMIC_RELAXED, __HIP_MEMORY_SCOPE_AGENT);
            return true;
        }
        return false;
    }
};

#define PACE16 232
__device__ __forceinline__ void pace(u32 t0, u32 slot) {
    const u32 due = (slot * (u32)PACE16) >> 4;
    while ((u32)__builtin_amdgcn_s_memrealtime() - t0 < due) __builtin_amdgcn_s_sleep(1);
}

#define STAMP(i) do {} while (0)

__global__ void __launch_bounds__(256, 1) gemm_rs_kernel(Params p) {
    extern __shared__ __attribute__((aligned(16))) char smem[];
    const int b = blockIdx.x;
    const int lane = threadIdx.x & 63;
    const int wave = __builtin_amdgcn_readfirstlane(threadIdx.x >> 6);
    const u32 rank = (u32)p.rank & 7u;
    char* me = p.ws[rank];
    const u32 lane16 = lane * 16;
    u32* ctr = (u32*)(me + WS_CTR + (u32)(b * 4 + wave) * 64u);

    u32 call = __hip_atomic_load(ctr, __ATOMIC_RELAXED, __HIP_MEMORY_SCOPE_AGENT) + 1u;
    if (b < N_PUSH) {
        constexpr int UPW = 128 / N_PUSH;
        if (wave < UPW) {
            const u32 u = (u32)b * UPW + (u32)wave;
            const rsrc_t rx = mk_rsrc(p.x, 2048u * 256u * 2u);
            const rsrc_t rw = mk_rsrc(p.w, 256u * 2048u * 2u);
            u32x4 xv[8], wv[8];
            const u32 xi = u & 1u, kh = (u >> 1) & 1u, xp = (u >> 2) & 3u, xm = u >> 4;
            {
                const u32 voff = (u32)(lane & 31) * 512u + (u32)(lane >> 5) * 16u;
                const u32 soff = (256u * xm + 128u * xi) * 512u + kh * 256u + xp * 64u;
#pragma unroll
                for (int f = 0; f < 8; ++f)
                    xv[f] = __builtin_amdgcn_raw_buffer_load_b128(rx, voff + (f & 1) * 32u, soff + (f >> 1) * 32u * 512u, 0);
            }
            const u32 wj = xi, ws3 = xp, wks = xm;
            {
                const u32 voff = (u32)(lane >> 2) * 4096u + (u32)(lane & 3) * 16u;
                const u32 soff = (128u * kh + 16u * wks) * 4096u + wj * 2048u + ws3 * 64u;
#pragma unroll
                for (int f = 0; f < 8; ++f)
                    wv[f] = __builtin_amdgcn_raw_buffer_load_b128(rw, voff + f * 256u, soff, 0);
            }
            char* const dx0 = p.ws[4u * kh + xi];
            char* const dx1 = p.ws[4u * kh + 2u + xi];
            char* const dw0 = p.ws[4u * kh + 2u * wj];
            char* const dw1 = p.ws[4u * kh + 2u * wj + 1u];
            __builtin_amdgcn_sched_barrier(0);
            call = __builtin_amdgcn_readfirstlane(call);
            const u32 ib = WS_IB + (call % 3u) * IB_BYTES;
            const u32 tp = (u32)__builtin_amdgcn_s_memrealtime();
            STAMP(1);
            {
                const rsrc_t r0 = mk_rsrc(dx0 + ib, IB_BYTES), r1 = mk_rsrc(dx1 + ib, IB_BYTES);
                const u32 soff = ((xm * 4u + (rank >> 1)) * 64u + (rank & 1u) * 8u + xp * 2u) * KIB;
#pragma unroll
                for (int f = 0; f < 8; ++f) {
                    const u32x4 v = nz4(xv[f]);
                    pace(tp, 4u * f + xp);
                    st16(v, r0, lane16 + (f & 1) * 1024u, soff + (f >> 1) * 16u * KIB);
                    st16(v, r1, lane16 + (f & 1) * 1024u, soff + (f >> 1) * 16u * KIB);
                    __builtin_amdgcn_sched_barrier(0);
                    if (f == 0) STAMP(2);
                }
            }
            STAMP(3);
            char* lw = smem + wave * 8 * KIB;
#pragma unroll
            for (int f = 0; f < 8; ++f) *LDSP(u32x4, lw + f * 1024 + lane16) = wv[f];
            const u32 troff = (u32)(8 * (lane >> 5) + ((lane & 15) >> 2)) * 64u + (u32)((lane >> 4) & 1) * 32u + (u32)(lane & 3) * 8u;
            {
                const rsrc_t r0 = mk_rsrc(dw0 + ib, IB_BYTES), r1 = mk_rsrc(dw1 + ib, IB_BYTES);
                const u32 soff = 2u * MIB + (ws3 * 64u + rank * 8u + wks) * KIB;
#pragma unroll
                for (int f = 0; f < 8; ++f) {
                    const s16x4 lo = __builtin_amdgcn_ds_read_tr16_b64_v4i16(LDSP(s16x4, lw + f * 1024 + troff));
                    const s16x4 hi = __builtin_amdgcn_ds_read_tr16_b64_v4i16(LDSP(s16x4, lw + f * 1024 + troff + 256));
                    const u32x2 l2 = __builtin_bit_cast(u32x2, lo), h2 = __builtin_bit_cast(u32x2, hi);
                    const u32x4 v = nz4(u32x4{l2.x, l2.y, h2.x, h2.y});
                    pace(tp, 32u + 4u * f + xp);
                    st16(v, r0, lane16, soff + f * 256u * KIB);
                    st16(v, r1, lane16, soff + f * 256u * KIB);
                }
            }
            STAMP(4);
        }
    }
    call = __builtin_amdgcn_readfirstlane(call);
    if (lane == 0) __hip_atomic_store(ctr, call, __ATOMIC_RELAXED, __HIP_MEMORY_SCOPE_AGENT);
    if (b < N_PUSH) { asm volatile("s_waitcnt vmcnt(0)" ::: "memory"); return; }
    Waiter wt{(u32*)(me + WS_TMO), call, __builtin_amdgcn_s_memrealtime(), 0u};
    const u32x4 zero = {0u, 0u, 0u, 0u};

    if (b < N_PUSH + N_GEMM) {
        const u32 m = (rank + 1u + (u32)b) & 7u;
        const u32 gi = (u32)(b - N_PUSH) >> 3;
        const bool pair = gi < N_PAIR;
        const u32 g = pair ? 2u * gi : gi + N_PAIR;
        const rsrc_t ri = mk_rsrc(me + WS_IB + (call % 3u) * IB_BYTES, IB_BYTES);
        const rsrc_t ro = mk_rsrc(p.ws[m] + WS_OB, 2u * MIB);
        bool dead = false;
        const u32 canoff = (u32)(lane & 15) * 1024u + 1020u;
#define PROBE16(base) do { \
            while (!dead) { \
                const u32 v_ = __builtin_amdgcn_raw_buffer_load_b32(ri, canoff, (base), AUX_SYS); \
                if (__all(v_ != 0u)) break; \
                if (wt.expired()) dead = true; \
                __builtin_amdgcn_s_sleep(GEMM_SLEEP); \
            } } while (0)
#define LOAD16(dst, base) do { \
            _Pragma("unroll") for (int f_ = 0; f_ < 16; ++f_) \
                dst[f_] = __builtin_amdgcn_raw_buffer_load_b128(ri, lane16 + (f_ & 3) * 1024u, (base) + (f_ >> 2) * 4096u, AUX_SYS); \
            } while (0)
#define CHECK16(dst, base) do { \
            for (;;) { \
                u32 mn_ = 0xffffffffu; \
                _Pragma("unroll") for (int f_ = 0; f_ < 16; ++f_) mn_ = min4(mn_, dst[f_]); \
                if (__all(mn_ != 0u) || dead) break; \
                if (wt.expired()) dead = true; \
                PROBE16(base); \
                LOAD16(dst, base); \
            } } while (0)
        const u32 xbase = (m * 4u + (u32)wave) * 64u * KIB;
        const u32 wbase0 = 2u * MIB + (g * 64u + (u32)wave * 16u) * KIB, wbase1 = pair ? wbase0 + 64u * KIB : wbase0;
        const rsrc_t ri1 = mk_rsrc(me + WS_IB + (call % 3u) * IB_BYTES, pair ? IB_BYTES : 0u);
        u32x4 xf0[16], xf1[16], xf2[16], xf3[16], wf[16], held[2];
        PROBE16(xbase); LOAD16(xf0, xbase); CHECK16(xf0, xbase);
        STAMP(1);
        PROBE16(xbase + 16u * KIB); LOAD16(xf1, xbase + 16u * KIB); CHECK16(xf1, xbase + 16u * KIB);
        PROBE16(xbase + 32u * KIB); LOAD16(xf2, xbase + 32u * KIB); CHECK16(xf2, xbase + 32u * KIB);
        PROBE16(xbase + 48u * KIB); LOAD16(xf3, xbase + 48u * KIB); CHECK16(xf3, xbase + 48u * KIB);
        STAMP(2);
        if (g >= OPT_G) PROBE16(wbase0);
        LOAD16(wf, wbase0);
        char* pw = smem + wave * 16 * KIB + (lane & 31) * 128;
        const u32 wsw = (u32)(lane >> 5) ^ (u32)(lane & 7);
        char* pr = smem + wave * 4 * KIB + (lane >> 2) * 128;
        const u32 rsw = (u32)(2 * (lane & 3)) ^ (u32)((lane >> 2) & 7);
        const u32 obase = ((rank * 32u + g) * 4u + (u32)wave) * 2048u;
#pragma unroll
        for (int st = 0; st < 2; ++st) {
            if (st == 1 && !pair) break;
            if (st == 0) CHECK16(wf, wbase0);
            if (dead) wf[0] = u32x4{0x7fc07fc0u, 0x7fc07fc0u, 0x7fc07fc0u, 0x7fc07fc0u};
            STAMP(3 + 4 * st);
#define PUT(rb, acc, G) *LDSP(f32x4, pw + st * 64 * KIB + (rb) * 4096 + ((wsw ^ (2u * (G))) << 4)) = f32x4{acc[4 * (G)], acc[4 * (G) + 1], acc[4 * (G) + 2], acc[4 * (G) + 3]}
#define TILE(rb, acc, xf, prev) do { \
                _Pragma("unroll") for (int kk = 0; kk < 16; ++kk) { \
                    acc = __builtin_amdgcn_mfma_f32_32x32x16_bf16(__builtin_bit_cast(bf16x8, wf[kk]), __builtin_bit_cast(bf16x8, xf[kk]), acc, 0, 0, 0); \
                    if ((rb) > 0 && kk >= 4 && kk < 12 && !(kk & 1)) PUT((rb) - 1, prev, (kk - 4) >> 1); \
                    if ((rb) == 3 && st == 0) \
                        wf[kk] = __builtin_amdgcn_raw_buffer_load_b128(ri1, lane16 + (kk & 3) * 1024u, wbase1 + (kk >> 2) * 4096u, AUX_SYS); \
                    __builtin_amdgcn_sched_barrier(0); \
                } } while (0)
            f32x16 acc0 = {}, acc1 = {}, acc2 = {}, acc3 = {};
            __builtin_amdgcn_sched_barrier(0);
            TILE(0, acc0, xf0, acc0); TILE(1, acc1, xf1, acc0); TILE(2, acc2, xf2, acc1); TILE(3, acc3, xf3, acc2);
            PUT(3, acc3, 0); PUT(3, acc3, 1); PUT(3, acc3, 2); PUT(3, acc3, 3);
            STAMP(4 + 4 * st);
            asm volatile("s_waitcnt lgkmcnt(0)" ::: "memory");
            __builtin_amdgcn_s_barrier();
            asm volatile("" ::: "memory");
            STAMP(5 + 4 * st);
            f32x4 t[16];
#pragma unroll
            for (int i = 0; i < 16; ++i)
                t[i] = *LDSP(f32x4, pr + st * 64 * KIB + ((i >> 1) & 3) * 16 * KIB + (i >> 3) * 2048 + ((rsw ^ (u32)(i & 1)) << 4));
            __builtin_amdgcn_sched_barrier(0);
#pragma unroll
            for (int e = 0; e < 2; ++e) {
                const f32x4 lo = (t[8 * e] + t[8 * e + 2]) + (t[8 * e + 4] + t[8 * e + 6]);
                const f32x4 hi = (t[8 * e + 1] + t[8 * e + 3]) + (t[8 * e + 5] + t[8 * e + 7]);
                held[e] = nz4(u32x4{pk_bf16(lo.x, lo.y), pk_bf16(lo.z, lo.w), pk_bf16(hi.x, hi.y), pk_bf16(hi.z, hi.w)});
                asm volatile("" : "+v"(held[e]));
            }
            if (st == 0 && pair) CHECK16(wf, wbase1);
            st16(held[0], ro, lane16, obase + st * 8192u);
            st16(held[1], ro, lane16 + 1024u, obase + st * 8192u);
            __builtin_amdgcn_sched_barrier(0);
            STAMP(6 + 4 * st);
        }
        asm volatile("s_waitcnt vmcnt(0)" ::: "memory");
        STAMP(11);
        return;
    }

    {
        const u32 idx = (u32)(b - RED0) * 4u + (u32)wave;
        {
            const rsrc_t rz = mk_rsrc(me + WS_IB + ((call + 2u) % 3u) * IB_BYTES, IB_BYTES);
            for (u32 n = idx; n < 4096u; n += N_RED * 4u)
                st16(zero, rz, lane16, n * KIB);
        }
        STAMP(6);
        const rsrc_t rb_ = mk_rsrc(me + WS_OB, 2u * MIB);
        const rsrc_t rout = mk_rsrc(p.out, 256u * 2048u * 2u);
        constexpr int NPC = 1024 / (N_RED * 4);
        constexpr u32 SG = 32 / NPC;
        static_assert(NPC * N_RED * 4 == 1024 && NPC <= 16, "reducer geometry");
        bool dead = false;
        const u32 quad = (idx >> 3) & 3u, pc = idx & 7u, grp = idx >> 5;
#define PS(j) (SG * (j) + ((grp + (j)) & (SG - 1u)))
#define POFF(j) ((quad * 32u + PS(j)) * 8192u + pc * 1024u)
        const u32 pj = ((u32)lane >> 1) & (NPC - 1u);
        const u32 probe = POFF(pj) + (u32)(lane & 1) * MIB + 1020u;
        const u32 ovoff = (u32)(lane >> 2) * 4096u + (u32)(lane & 3) * 16u + (128u * (quad & 1u) + 16u * pc) * 4096u + (quad >> 1) * 2048u;
        u32 pending = (1u << NPC) - 1u;
        while (pending) {
            u32 act = pending;
            if (__builtin_popcount(pending) > RED_PAR && !dead) {
                const u32 v = __builtin_amdgcn_raw_buffer_load_b32(rb_, probe, 0u, AUX_SYS);
                u32 got = (u32)__ballot(v != 0u);
                got &= got >> 1;
                act = 0;
#pragma unroll
                for (int j = 0; j < NPC; ++j) act |= (got >> (2 * j) & 1u) << j;
                act &= pending;
                if (!act) {
                    if (wt.expired()) dead = true;
                    __builtin_amdgcn_s_sleep(RED_SLEEP);
                    continue;
                }
            }
            u32x4 v0[NPC], v1[NPC];
#pragma unroll
            for (int j = 0; j < NPC; ++j) if (act >> j & 1u) {
                v0[j] = __builtin_amdgcn_raw_buffer_load_b128(rb_, lane16, POFF(j), AUX_SYS);
                v1[j] = __builtin_amdgcn_raw_buffer_load_b128(rb_, lane16, POFF(j) + 1u * MIB, AUX_SYS);
            }
            __builtin_amdgcn_s_waitcnt(0x0F70);
#pragma unroll
            for (int j = 0; j < NPC; ++j) if (act >> j & 1u) {
                const u32 mn = min4(min4(0xffffffffu, v0[j]), v1[j]);
                if (__all(mn != 0u) || dead) {
                    u32x4 r = {add_relu(v0[j].x, v1[j].x), add_relu(v0[j].y, v1[j].y), add_relu(v0[j].z, v1[j].z), add_relu(v0[j].w, v1[j].w)};
                    if (dead) r = u32x4{0x7fc07fc0u, 0x7fc07fc0u, 0x7fc07fc0u, 0x7fc07fc0u};
                    st16(r, rout, ovoff, PS(j) * 64u);
                    st16(zero, rb_, lane16, POFF(j));
                    st16(zero, rb_, lane16, POFF(j) + 1u * MIB);
                    pending &= ~(1u << j);
                    STAMP(7 + j * 4 / NPC);
                }
            }
            if (pending && wt.expired()) dead = true;
        }
        asm volatile("s_waitcnt vmcnt(0)" ::: "memory");
        STAMP(11);
    }
}

extern "C" void kernel_launch(int rank, int nranks, ncclComm_t comm, hipStream_t stream,
                              void* const* d_in, const int* in_sizes, int n_in,
                              void* d_out, int out_size,
                              const int* in_dtypes, int out_dtype,
                              void* const* d_out_all,
                              void* const* d_ws_all, size_t ws_bytes) {
    if (nranks != NR || (unsigned)rank >= (unsigned)NR || n_in != 2 || ws_bytes < WS_NEED ||
        in_sizes[0] != 2048 * 256 || in_sizes[1] != 256 * 2048 || out_size != 256 * 2048) {
        fprintf(stderr, "kernel_launch: unexpected problem shape\n");
        return;
    }
    static bool attr_set = false;
    if (!attr_set) {
        hipError_t e = hipFuncSetAttribute((const void*)gemm_rs_kernel, hipFuncAttributeMaxDynamicSharedMemorySize, LDS_BYTES);
        if (e != hipSuccess) { fprintf(stderr, "hipFuncSetAttribute: %s\n", hipGetErrorString(e)); return; }
        attr_set = true;
    }
    Params p;
    for (int r = 0; r < NR; ++r) p.ws[r] = (char*)d_ws_all[r];
    p.x = (const char*)d_in[0];
    p.w = (const char*)d_in[1];
    p.out = (char*)d_out;
    p.rank = rank;
    gemm_rs_kernel<<<N_WG, 256, LDS_BYTES, stream>>>(p);
}
```

```cpp
#include <hip/hip_runtime.h>
#include <rccl/rccl.h>
#include <cstdio>

typedef unsigned u32;
typedef unsigned long long u64;
typedef u32 u32x4 __attribute__((ext_vector_type(4)));
typedef u32 u32x2 __attribute__((ext_vector_type(2)));
typedef short s16x4 __attribute__((ext_vector_type(4)));
typedef __bf16 bf16x8 __attribute__((ext_vector_type(8)));
typedef __bf16 bf16x2 __attribute__((ext_vector_type(2)));
typedef float f32x2 __attribute__((ext_vector_type(2)));
typedef float f32x16 __attribute__((ext_vector_type(16)));
typedef float f32x4 __attribute__((ext_vector_type(4)));
typedef __amdgpu_buffer_rsrc_t rsrc_t;
typedef __attribute__((address_space(1))) u32 gu32;
#define LDSP(T, p) ((__attribute__((address_space(3))) T*)(p))

constexpr int NR = 8;
constexpr u32 KIB = 1024u, MIB = 1024u * 1024u;
constexpr u32 WS_CTR = 0;
constexpr u32 WS_TMO = 64 * KIB;
constexpr u32 WS_LOG = 128 * KIB;
constexpr u32 WS_OB = 1 * MIB;
constexpr u32 WS_IB = 4 * MIB;
constexpr u32 IB_BYTES = 4 * MIB;
constexpr u32 WS_NEED = WS_IB + 3 * IB_BYTES;
#define NPUSH 64
#define NGEMM 192
constexpr int N_PUSH = NPUSH, N_GEMM = NGEMM, N_WG = 256;
constexpr bool FUSE_PR = N_PUSH + N_GEMM == N_WG;
constexpr int N_RED = FUSE_PR ? N_PUSH : N_WG - N_PUSH - N_GEMM, RED0 = FUSE_PR ? 0 : N_PUSH + N_GEMM;
constexpr u32 N_PAIR = 32 - N_GEMM / 8;
constexpr u32 LDS_BYTES = 128 * KIB;
#define RED_B 4
#define RED_PAR 3
#define OPT_G 4u
#define GEMM_SLEEP "8"
#define RED_SLEEP 4
constexpr u64 DEADLINE_TICKS = 20000000ull;
constexpr int AUX_SYS = 17;

struct Params {
    char* ws[NR];
    const char* x;
    const char* w;
    char* out;
    char* me;
    int rank;
};

__device__ __forceinline__ rsrc_t mk_rsrc(const void* p, u32 bytes) {
    return __builtin_amdgcn_make_buffer_rsrc((void*)p, 0, bytes, 0x00020000);
}
__device__ __forceinline__ void st16(u32x4 v, rsrc_t r, u32 voff, u32 soff) {
    __builtin_amdgcn_raw_buffer_store_b128(v, r, voff + soff, 0, AUX_SYS);
}
__device__ __forceinline__ u32 nz(u32 d) { return d ? d : 0x80000000u; }
__device__ __forceinline__ u32x4 nz4(u32x4 v) { return u32x4{nz(v.x), nz(v.y), nz(v.z), nz(v.w)}; }
__device__ __forceinline__ u32 min4(u32 m, u32x4 v) { return min(min(m, min(v.x, v.y)), min(v.z, v.w)); }
__device__ __forceinline__ void min4_pinned(u32& m, u32x4 v) {
    asm volatile("v_min3_u32 %0, %0, %1, %2\n\tv_min3_u32 %0, %0, %3, %4" : "+v"(m) : "v"(v.x), "v"(v.y), "v"(v.z), "v"(v.w));
}
__device__ __forceinline__ u32 pk_bf16(float a, float b) {
    return __builtin_bit_cast(u32, __builtin_convertvector(f32x2{a, b}, bf16x2));
}
__device__ __forceinline__ float bf_lo(u32 d) { return __builtin_bit_cast(float, d << 16); }
__device__ __forceinline__ float bf_hi(u32 d) { return __builtin_bit_cast(float, d & 0xffff0000u); }
__device__ __forceinline__ float relu(float v) { return v < 0.0f ? 0.0f : v; }
__device__ __forceinline__ u32 add_relu(u32 a, u32 b, u32 c) {
    return pk_bf16(relu(bf_lo(a) + bf_lo(b) + bf_lo(c)), relu(bf_hi(a) + bf_hi(b) + bf_hi(c)));
}

__device__ __forceinline__ bool probe_spin(u32x4 rs, u32 voff, u32 soff, u32& a, u32& b) {
    u32 n;
    asm volatile(
        "s_nop 4\n\t"
        "buffer_load_dword %0, %3, %4, %5 offen sc0 sc1\n\t"
        "s_sleep " GEMM_SLEEP "\n\t"
        "buffer_load_dword %1, %3, %4, %5 offen sc0 sc1\n\t"
        "s_movk_i32 %2, 64\n"
        "1:\n\t"
        "s_waitcnt vmcnt(1)\n\t"
        "v_cmp_ne_u32 vcc, 0, %0\n\t"
        "s_cmp_eq_u64 vcc, exec\n\t"
        "s_cbranch_scc1 2f\n\t"
        "buffer_load_dword %0, %3, %4, %5 offen sc0 sc1\n\t"
        "s_waitcnt vmcnt(1)\n\t"
        "v_cmp_ne_u32 vcc, 0, %1\n\t"
        "s_cmp_eq_u64 vcc, exec\n\t"
        "s_cbranch_scc1 2f\n\t"
        "buffer_load_dword %1, %3, %4, %5 offen sc0 sc1\n\t"
        "s_sub_u32 %2, %2, 1\n\t"
        "s_cmp_lg_u32 %2, 0\n\t"
        "s_cbranch_scc1 1b\n"
        "2:"
        : "=&v"(a), "=&v"(b), "=&s"(n) : "v"(voff), "s"(rs), "s"(soff) : "vcc", "scc", "memory");
    return n != 0u;
}

struct Waiter {
    gu32* tmo; u32 call; u64 t0; u32 spins;
    __device__ __forceinline__ bool expired() {
        asm volatile("" ::: "memory");
        if ((++spins & 63u) != 0) return false;
        return expired_now();
    }
    __device__ __forceinline__ bool expired_now() {
        u32 t = __hip_atomic_load(tmo, __ATOMIC_RELAXED, __HIP_MEMORY_SCOPE_AGENT);
        t = __builtin_amdgcn_readfirstlane(t);
        if (t == call || __builtin_amdgcn_s_memrealtime() - t0 > DEADLINE_TICKS) {
            __hip_atomic_store(tmo, call, __ATOMIC_RELAXED, __HIP_MEMORY_SCOPE_AGENT);
            return true;
        }
        return false;
    }
};

#define PACE16 232
#define LEAD 90
#define ZPACE 7u
#define AUX_BULK 17
#define UMAP 1
#define DIAG 1
#define DIAG_DELAY 32
#define OPACE16 224
#define OLEAD 0
#define GATE 16
#define RGATE 0
#define WARM 0
__device__ __forceinline__ void pace(u32 t0, u32 kib) {
    const u32 due = (u32)LEAD + ((kib * (u32)PACE16) >> 7);
    while ((u32)__builtin_amdgcn_s_memrealtime() - t0 < due) __builtin_amdgcn_s_sleep(1);
}

#define STAMP(i) do {} while (0)
#define STAMPC(i) do {} while (0)
#define STAMP_FLUSH() do {} while (0)

constexpr u32 XS = DIAG ? 7u : 8u;
__device__ __forceinline__ void pace_out(u32 t0, u32 idx) {
    const u32 due = (u32)(LEAD + OLEAD) + (((32u * XS + 256u) * (u32)PACE16) >> 7) + ((idx * (u32)OPACE16) >> 4);
    if (OPACE16) while ((u32)__builtin_amdgcn_s_memrealtime() - t0 < due) __builtin_amdgcn_s_sleep(1);
}

__global__ void __launch_bounds__(256, 1) gemm_rs_kernel(Params p) {
    extern __shared__ __attribute__((aligned(16))) char smem[];
    const u64 t_start = __builtin_amdgcn_s_memrealtime();
    const int b = blockIdx.x;
    const int lane = threadIdx.x & 63;
    const int wave = __builtin_amdgcn_readfirstlane(threadIdx.x >> 6);
    const u32 lane16 = lane * 16;
    constexpr int UPW = 128 / N_PUSH;
    const u32 u = ((u32)b + (u32)wave * N_PUSH) & 127u;
    const u32 xi = u & 1u, kh = (u >> 1) & 1u, xp = (u >> 2) & 3u, xm = u >> 4;
    u32 rank = (u32)p.rank;
    char* me = p.me;
    const char* px = p.x; const char* pw_ = p.w; char* pout = p.out;
    char* d0 = p.ws[b < N_PUSH ? 4u * kh + xi : (u32)b & 7u];
    char* d1 = p.ws[4u * kh + 2u + xi];
    char* d2 = p.ws[4u * kh + 2u * xi];
    char* d3 = p.ws[4u * kh + 2u * xi + 1u];
    asm volatile("" : "+s"(rank), "+s"(me), "+s"(px), "+s"(pw_), "+s"(pout), "+s"(d0), "+s"(d1), "+s"(d2), "+s"(d3));
    rank &= 7u;
    gu32* ctr = (gu32*)(me + WS_CTR + (u32)(b * 4 + wave) * 64u);

    u32 call = __hip_atomic_load(ctr, __ATOMIC_RELAXED, __HIP_MEMORY_SCOPE_AGENT);
    if (!FUSE_PR && b < N_PUSH && wave >= UPW) return;
    if (__builtin_expect(b < N_PUSH && wave < UPW, 1)) {
        const rsrc_t rx = mk_rsrc(px, 2048u * 256u * 2u);
        const rsrc_t rw = mk_rsrc(pw_, 256u * 2048u * 2u);
        u32x4 xv[8], wv[8];
        {
            const u32 voff = (u32)(lane & 31) * 512u + (u32)(lane >> 5) * 16u;
            const u32 soff = (256u * xm + 128u * xi) * 512u + kh * 256u + xp * 64u;
#pragma unroll
            for (int f = 0; f < 8; ++f)
                xv[f] = __builtin_amdgcn_raw_buffer_load_b128(rx, voff + (f & 1) * 32u, soff + (f >> 1) * 32u * 512u, 0);
        }
        {
            const u32 voff = (u32)(lane >> 2) * 4096u + (u32)(lane & 3) * 16u;
            const u32 soff = (128u * kh + 16u * xm) * 4096u + xi * 2048u + xp * 64u;
#pragma unroll
            for (int f = 0; f < 8; ++f)
                wv[f] = __builtin_amdgcn_raw_buffer_load_b128(rw, voff + f * 256u, soff, 0);
        }
        __builtin_amdgcn_sched_barrier(0);
        const u32 ib = WS_IB + ((__builtin_amdgcn_readfirstlane(call) + 1u) % 3u) * IB_BYTES;
        const u32 tp = (u32)t_start;
        STAMP(1);
        if (!DIAG || xm != rank) {
            const rsrc_t r0 = mk_rsrc(d0 + ib, IB_BYTES), r1 = mk_rsrc(d1 + ib, IB_BYTES);
            const u32 soff = ((xm * 4u + (rank >> 1)) * 64u + (rank & 1u) * 8u + xp * 2u) * KIB;
#pragma unroll
            for (int f = 0; f < 8; ++f) {
                const u32x4 v = nz4(xv[f]);
                if (f == 0) { asm volatile("" :: "v"(v)); STAMP(2); }
                pace(tp, (4u * f + xp) * XS);
                st16(v, r0, lane16 + (f & 1) * 1024u, soff + (f >> 1) * 16u * KIB);
                st16(v, r1, lane16 + (f & 1) * 1024u, soff + (f >> 1) * 16u * KIB);
                __builtin_amdgcn_sched_barrier(0);
            }
        }
        STAMP(3);
        char* lw = smem + wave * 8 * KIB;
#pragma unroll
        for (int f = 0; f < 8; ++f) *LDSP(u32x4, lw + f * 1024 + lane16) = wv[f];
        const u32 troff = (u32)(8 * (lane >> 5) + ((lane & 15) >> 2)) * 64u + (u32)((lane >> 4) & 1) * 32u + (u32)(lane & 3) * 8u;
        {
            const rsrc_t r0 = mk_rsrc(d2 + ib, IB_BYTES), r1 = mk_rsrc(d3 + ib, IB_BYTES);
            const u32 soff = 2u * MIB + (xp * 64u + rank * 8u + xm) * KIB;
#pragma clang loop unroll(disable)
            for (u32 f = 0; f < 8; ++f) {
                const s16x4 lo = __builtin_amdgcn_ds_read_tr16_b64_v4i16(LDSP(s16x4, lw + f * 1024 + troff));
                const s16x4 hi = __builtin_amdgcn_ds_read_tr16_b64_v4i16(LDSP(s16x4, lw + f * 1024 + troff + 256));
                const u32x2 l2 = __builtin_bit_cast(u32x2, lo), h2 = __builtin_bit_cast(u32x2, hi);
                const u32x4 v = nz4(u32x4{l2.x, l2.y, h2.x, h2.y});
                pace(tp, 32u * XS + (4u * f + xp) * 8u);
                st16(v, r0, lane16, soff + f * 256u * KIB);
                st16(v, r1, lane16, soff + f * 256u * KIB);
            }
        }
        STAMP(4);
    }
    call = __builtin_amdgcn_readfirstlane(call) + 1u;
    if (lane == 0) __hip_atomic_store(ctr, call, __ATOMIC_RELAXED, __HIP_MEMORY_SCOPE_AGENT);
    if (!FUSE_PR && b < N_PUSH) { asm volatile("s_waitcnt vmcnt(0)" ::: "memory"); STAMP_FLUSH(); return; }
    Waiter wt{(gu32*)(me + WS_TMO), call, t_start, 0u};
    const u32x4 zero = {0u, 0u, 0u, 0u};
    const u32x4 nanv = {0x7fc07fc0u, 0x7fc07fc0u, 0x7fc07fc0u, 0x7fc07fc0u};

    if (b >= N_PUSH && b < N_PUSH + N_GEMM) {
        const u32 m = (u32)b & 7u;
        const u32 gi = (u32)(b - N_PUSH) >> 3;
        const bool pair = gi < N_PAIR;
        const u32 g = pair ? 2u * gi : gi + N_PAIR;
        char* const ibp = me + WS_IB + (call % 3u) * IB_BYTES;
        if (FUSE_PR && UPW != 2) {
            const rsrc_t rz = mk_rsrc(me + WS_IB + ((call + 2u) % 3u) * IB_BYTES, IB_BYTES);
            for (u32 n = (u32)(b - N_PUSH) * 4u + (u32)wave; n < 4096u; n += N_GEMM * 4u) st16(zero, rz, lane16, n * KIB);
        }
        const u32 canw = (u32)(lane & 15) * 1024u + 1020u;
        const bool hole = DIAG && (u32)wave == (m >> 1), skip_lo = hole && !(m & 1u), skip_hi = hole && (m & 1u);
        const u32 can = skip_lo ? canw | 8192u : skip_hi ? canw & ~8192u : canw;
        const rsrc_t rlo = mk_rsrc(ibp, skip_lo ? 0u : IB_BYTES), rhi = mk_rsrc(ibp, skip_hi ? 0u : IB_BYTES);
#define ALLIN(lo_, hi_) ((__all((lo_) != 0u) || skip_lo) && (__all((hi_) != 0u) || skip_hi))
        const rsrc_t ri = mk_rsrc(ibp, IB_BYTES);
        const rsrc_t ro = mk_rsrc(d0 + WS_OB, 2u * MIB);
        bool dead = false;
#define PROBE1(base) __builtin_amdgcn_raw_buffer_load_b32(ri, can, (base), AUX_SYS)
        const u32x4 riw = {(u32)(size_t)ibp, (u32)((size_t)ibp >> 32) & 0xffffu, IB_BYTES, 0x00020000u};
        u32 pa = 0, pb = 0;
#define PROBE16(base) do { while (!dead && !probe_spin(riw, can, (base), pa, pb)) dead = wt.expired_now(); } while (0)
#define KEEP() asm volatile("" :: "v"(pa), "v"(pb))
#define LOADF(base, f_) __builtin_amdgcn_raw_buffer_load_b128((f_) < 8 ? rlo : rhi, lane16 + ((f_) & 3) * 1024u, (base) + ((f_) >> 2) * 4096u, AUX_SYS)
#define SB() __builtin_amdgcn_sched_barrier(0)
#define LOAD16(dst, base) do { _Pragma("unroll") for (int f_ = 0; f_ < 16; ++f_) { dst[f_] = LOADF(base, f_); asm volatile("" ::: "memory"); SB(); } } while (0)
#define FETCH16(dst, base, next, hit, pin) do { \
            for (;;) { \
                if (!hit) PROBE16(base); \
                LOAD16(dst, base); \
                u32 nx_ = PROBE1(next); \
                u32 lo_ = 0xffffffffu, hi_ = 0xffffffffu; \
                _Pragma("unroll") for (int f_ = 0; f_ < 16; ++f_) { min4_pinned(f_ < 8 ? lo_ : hi_, dst[f_]); if (pin) asm volatile("" : "+a"(dst[f_])); SB(); } \
                KEEP(); \
                asm volatile("" : "+v"(nx_)); \
                hit = __all(nx_ != 0u); \
                if (ALLIN(lo_, hi_) || dead) break; \
                hit = false; \
                if (wt.expired()) dead = true; \
            } \
            } while (0)
        const u32 xbase = (m * 4u + (u32)wave) * 64u * KIB;
        const u32 wbase = 2u * MIB + (g * 64u + (u32)wave * 16u) * KIB;
        u32x4 xf0[16], xf1[16], xf2[16], xf3[16], wf[16], ov[2];
#define LKEY(r) ((((r) >> 1) & 1u) | (((r) & 1u) << 1) | ((r) & 4u))
        char* const pw = smem + (u32)wave * 16u * KIB + (u32)(lane & 31) * 128u;
        const u32 wsw = (u32)(lane >> 5) ^ LKEY((u32)lane);
        char* const pr = smem + (u32)wave * 4u * KIB + (u32)(lane >> 2) * 128u;
        const u32 rsw = (u32)(2 * (lane & 3)) ^ LKEY((u32)lane >> 2);
        const u32 obase = ((rank * 32u + g) * 4u + (u32)wave) * 2048u;
#define MMA(acc, a_, b_) do { acc = __builtin_amdgcn_mfma_f32_32x32x16_bf16(__builtin_bit_cast(bf16x8, a_), __builtin_bit_cast(bf16x8, b_), acc, 0, 0, 0); \
                              asm volatile("" : "+a"(acc)); } while (0)
#define PUT(st, rb, acc, G) *LDSP(f32x4, pw + (st) * 64 * KIB + (rb) * 4096 + ((wsw ^ (2u * (G))) << 4)) = f32x4{acc[4 * (G)], acc[4 * (G) + 1], acc[4 * (G) + 2], acc[4 * (G) + 3]}
#define COMPUTE(st, pref) do { \
            f32x16 acc0 = {}, acc1 = {}, acc2 = {}, acc3 = {}; \
            SB(); \
            _Pragma("unroll") for (int kk = 0; kk < 8; ++kk) { \
                MMA(acc0, wf[kk], xf0[kk]); if (st == 0) min4_pinned(mlo, wf[kk]); SB(); \
                MMA(acc1, wf[kk], xf1[kk]); MMA(acc2, wf[kk], xf2[kk]); MMA(acc3, wf[kk], xf3[kk]); SB(); \
            } \
            _Pragma("unroll") for (int kk = 8; kk < 16; ++kk) { MMA(acc0, wf[kk], xf0[kk]); if (st == 0) min4_pinned(mhi, wf[kk]); SB(); } \
            _Pragma("unroll") for (int kk = 8; kk < 16; ++kk) { MMA(acc1, wf[kk], xf1[kk]); if (kk >= 11 && kk < 15) PUT(st, 0, acc0, kk - 11); \
                if (pref) wf[kk - 8] = LOADF(wbase + 64u * KIB, kk - 8); SB(); } \
            _Pragma("unroll") for (int kk = 8; kk < 16; ++kk) { MMA(acc2, wf[kk], xf2[kk]); if (kk >= 11 && kk < 15) PUT(st, 1, acc1, kk - 11); SB(); } \
            _Pragma("unroll") for (int kk = 8; kk < 16; ++kk) { MMA(acc3, wf[kk], xf3[kk]); if (kk >= 11 && kk < 15) PUT(st, 2, acc2, kk - 11); \
                if (pref) wf[kk] = LOADF(wbase + 64u * KIB, kk); SB(); } \
            PUT(st, 3, acc3, 0); PUT(st, 3, acc3, 1); PUT(st, 3, acc3, 2); PUT(st, 3, acc3, 3); \
            } while (0)
#define POISON(st) do { if (dead) { f32x16 q; for (int i = 0; i < 16; ++i) q[i] = __builtin_nanf(""); \
            for (int rb = 0; rb < 4; ++rb) for (int G = 0; G < 4; ++G) PUT(st, rb, q, G); } } while (0)
#define SUMUP(st, v) do { \
            asm volatile("s_waitcnt lgkmcnt(0)" ::: "memory"); \
            __builtin_amdgcn_s_barrier(); \
            asm volatile("" ::: "memory"); \
            STAMP(5 + 4 * (st)); \
            f32x4 t[16]; \
            _Pragma("unroll") for (int i = 0; i < 16; ++i) \
                t[i] = *LDSP(f32x4, pr + (st) * 64 * KIB + ((i >> 1) & 3) * 16 * KIB + (i >> 3) * 2048 + ((rsw ^ (u32)(i & 1)) << 4)); \
            SB(); \
            _Pragma("unroll") for (int e = 0; e < 2; ++e) { \
                const f32x4 lo = (t[8 * e] + t[8 * e + 2]) + (t[8 * e + 4] + t[8 * e + 6]); \
                const f32x4 hi = (t[8 * e + 1] + t[8 * e + 3]) + (t[8 * e + 5] + t[8 * e + 7]); \
                v[e] = nz4(u32x4{pk_bf16(lo.x, lo.y), pk_bf16(lo.z, lo.w), pk_bf16(hi.x, hi.y), pk_bf16(hi.z, hi.w)}); \
            } } while (0)
        if (DIAG && gi >= N_GEMM / 8 - 16) {
            const u32 dt = (gi - (N_GEMM / 8 - 16)) * 8u + m, dq = dt >> 5, ds = dt & 31u;
            const rsrc_t rx = mk_rsrc(px, 2048u * 256u * 2u);
            const rsrc_t rw = mk_rsrc(pw_, 256u * 2048u * 2u);
            if (DIAG_DELAY) __builtin_amdgcn_s_sleep(DIAG_DELAY);
            u32x4 wv[4];
#pragma unroll
            for (int c = 0; c < 4; ++c)
                wv[c] = __builtin_amdgcn_raw_buffer_load_b128(rw, (u32)(lane >> 2) * 4096u + (u32)(lane & 3) * 16u,
                                                              (64u * (u32)wave + 16u * c) * 4096u + (dq >> 1) * 2048u + ds * 64u, 0);
#pragma unroll
            for (int f = 0; f < 16; ++f)
                wf[f] = __builtin_amdgcn_raw_buffer_load_b128(rx, (u32)(lane & 31) * 512u + (u32)(lane >> 5) * 16u + (f & 3) * 32u,
                                                              (256u * rank + 128u * (dq & 1u) + 32u * (f >> 2)) * 512u + (u32)wave * 128u, 0);
            char* lw = smem + wave * 16 * KIB;
            const u32 troff = (u32)(8 * (lane >> 5) + ((lane & 15) >> 2)) * 64u + (u32)((lane >> 4) & 1) * 32u + (u32)(lane & 3) * 8u;
#pragma unroll
            for (int c = 0; c < 4; ++c) *LDSP(u32x4, lw + c * 1024 + lane16) = wv[c];
#pragma unroll
            for (int c = 0; c < 4; ++c) {
                const u32x2 l2 = __builtin_bit_cast(u32x2, __builtin_amdgcn_ds_read_tr16_b64_v4i16(LDSP(s16x4, lw + c * 1024 + troff)));
                const u32x2 h2 = __builtin_bit_cast(u32x2, __builtin_amdgcn_ds_read_tr16_b64_v4i16(LDSP(s16x4, lw + c * 1024 + troff + 256)));
                wv[c] = u32x4{l2.x, l2.y, h2.x, h2.y};
            }
#pragma unroll
            for (int rb = 0; rb < 4; ++rb) {
                f32x16 acc = {};
#pragma unroll
                for (int c = 0; c < 4; ++c) MMA(acc, wv[c], wf[rb * 4 + c]);
                PUT(1, rb, acc, 0); PUT(1, rb, acc, 1); PUT(1, rb, acc, 2); PUT(1, rb, acc, 3);
            }
            SUMUP(1, ov);
            const rsrc_t rd = mk_rsrc(me + WS_OB + 2u * MIB, 1u * MIB);
            st16(ov[0], rd, lane16, (dt * 4u + (u32)wave) * 2048u);
            st16(ov[1], rd, lane16 + 1024u, (dt * 4u + (u32)wave) * 2048u);
            STAMP(14);
        }
        bool hit = false;
        if (GATE) pace((u32)t_start, 7u * XS + GATE);
        FETCH16(xf0, xbase, xbase + 16u * KIB, hit, true);
        STAMP(1);
        if (GATE) pace((u32)t_start, 15u * XS + GATE);
        FETCH16(xf1, xbase + 16u * KIB, xbase + 32u * KIB, hit, true);
        if (GATE) pace((u32)t_start, 23u * XS + GATE);
        FETCH16(xf2, xbase + 32u * KIB, xbase + 48u * KIB, hit, true);
        if (GATE) pace((u32)t_start, 31u * XS + GATE);
        FETCH16(xf3, xbase + 48u * KIB, wbase, hit, false);
        if (GATE) pace((u32)t_start, 32u * XS + g * 8u + GATE);
        STAMP(2);
        for (;;) {
            if (!hit) PROBE16(wbase);
            LOAD16(wf, wbase);
            STAMP(3); STAMPC(12);
            u32 mlo = 0xffffffffu, mhi = 0xffffffffu;
            if (pair) COMPUTE(0, true); else COMPUTE(0, false);
            KEEP();
            if (ALLIN(mlo, mhi) || dead) break;
            hit = false;
            if (wt.expired()) dead = true;
        }
        POISON(0);
        STAMPC(13); STAMP(4);
        SUMUP(0, ov);
        {
            const u32 j = gi - N_PAIR;
            if (m != rank) pace_out((u32)t_start, pair ? gi : j < 7u ? 8u + j : j < 15u ? 2u * j + 2u : 31u);
        }
        st16(ov[0], ro, lane16, obase);
        st16(ov[1], ro, lane16 + 1024u, obase);
        SB();
        STAMP(6);
        if (pair) {
            u32 mlo = 0xffffffffu, mhi = 0xffffffffu;
#pragma unroll
            for (int f = 0; f < 16; ++f) min4_pinned(f < 8 ? mlo : mhi, wf[f]);
            if (!ALLIN(mlo, mhi)) { hit = false; FETCH16(wf, wbase + 64u * KIB, wbase, hit, false); }
            STAMP(7);
            COMPUTE(1, false);
            POISON(1);
            STAMP(8);
            SUMUP(1, ov);
            if (m != rank) pace_out((u32)t_start, 15u + 2u * gi);
            st16(ov[0], ro, lane16, obase + 8192u);
            st16(ov[1], ro, lane16 + 1024u, obase + 8192u);
            STAMP(10);
        }
        asm volatile("s_waitcnt vmcnt(0)" ::: "memory");
        STAMP(11);
        STAMP_FLUSH();
        return;
    }

    {
        const u32 idx = (u32)(b - RED0) * 4u + (u32)wave;
        if (!FUSE_PR) {
            const rsrc_t rz = mk_rsrc(me + WS_IB + ((call + 2u) % 3u) * IB_BYTES, IB_BYTES);
            for (u32 n = idx; n < 4096u; n += N_RED * 4u) st16(zero, rz, lane16, n * KIB);
        }
        if (FUSE_PR && UPW == 2 && wave >= 2) {
            const rsrc_t rz = mk_rsrc(me + WS_IB + ((call + 2u) % 3u) * IB_BYTES, IB_BYTES);
            const u32 z = (u32)b * 2u + (u32)wave - 2u;
#pragma clang loop unroll(disable)
            for (u32 n = 0; n < 32u; ++n) { pace((u32)t_start, n * ZPACE * 2u); st16(zero, rz, lane16, (n * 128u + z) * KIB); }
        }
        STAMP(6);
        constexpr int NPC = 1024 / (N_RED * 4);
        constexpr u32 SG = 32 / NPC;
        static_assert(NPC * N_RED * 4 == 1024 && NPC <= 16, "reducer geometry");
        const u32 quad = (idx >> 3) & 3u, pc = idx & 7u, grp = idx >> 5;
#define PS(j) (SG * (j) + ((grp + (j)) & (SG - 1u)))
#define POFF(j) ((quad * 32u + PS(j)) * 8192u + pc * 1024u)
        const u32 pj = ((u32)lane >> 1) & (NPC - 1u);
        const u32 probe = POFF(pj) + (u32)(lane & 1) * MIB + 1020u;
        const u32 ovoff = (u32)(lane >> 2) * 4096u + (u32)(lane & 3) * 16u + (128u * (quad & 1u) + 16u * pc) * 4096u + (quad >> 1) * 2048u;
        if (RGATE) pace_out((u32)t_start, RGATE);
        bool dead = false, dry = WARM;
        u32 pending = (1u << NPC) - 1u;
        { u32 q = 0; for (int j = 0; j < NPC; ++j) q |= 1u << (2 * j); pending = q; }
        while (pending) {
            const rsrc_t rb_ = mk_rsrc(me + WS_OB, dry ? 0u : 3u * MIB);
            const rsrc_t rout = mk_rsrc(pout, dry ? 0u : 256u * 2048u * 2u);
            u32 act = pending;
            if (__builtin_popcount(pending) > RED_PAR && !dead && !dry) {
                const u32 v = __builtin_amdgcn_raw_buffer_load_b32(rb_, probe, 0u, AUX_SYS);
                u32 got = (u32)__ballot(v != 0u);
                act = got & (got >> 1) & pending;
                if (!act) {
                    if (wt.expired()) dead = true;
                    __builtin_amdgcn_s_sleep(RED_SLEEP);
                    continue;
                }
            }
            u32x4 v0[RED_B], v1[RED_B], v2[RED_B];
            u32 bit[RED_B], pcs[RED_B];
#pragma unroll
            for (int c = 0; c < RED_B; ++c) {
                bit[c] = act & (0u - act); act ^= bit[c];
                const u32 j = bit[c] ? (u32)__builtin_ctz(bit[c]) >> 1 : 0u;
                pcs[c] = PS(j);
                if (bit[c]) {
                    const u32 off = (quad * 32u + pcs[c]) * 8192u + pc * 1024u;
                    v0[c] = __builtin_amdgcn_raw_buffer_load_b128(rb_, lane16, off, AUX_SYS);
                    v1[c] = __builtin_amdgcn_raw_buffer_load_b128(rb_, lane16, off + 1u * MIB, AUX_SYS);
                    if (DIAG) v2[c] = __builtin_amdgcn_raw_buffer_load_b128(rb_, lane16, off + 2u * MIB, AUX_SYS);
                    else v2[c] = u32x4{0x80008000u, 0x80008000u, 0x80008000u, 0x80008000u};
                }
            }
            __builtin_amdgcn_s_waitcnt(0x0F70);
#pragma unroll
            for (int c = 0; c < RED_B; ++c) if (bit[c]) {
                const u32 mn = min4(min4(min4(0xffffffffu, v0[c]), v1[c]), v2[c]);
                if (__all(mn != 0u) || dead || dry) {
                    const u32 off = (quad * 32u + pcs[c]) * 8192u + pc * 1024u;
                    u32x4 r = {add_relu(v0[c].x, v1[c].x, v2[c].x), add_relu(v0[c].y, v1[c].y, v2[c].y), add_relu(v0[c].z, v1[c].z, v2[c].z), add_relu(v0[c].w, v1[c].w, v2[c].w)};
                    if (dead) r = nanv;
                    st16(r, rout, ovoff, pcs[c] * 64u);
                    st16(zero, rb_, lane16, off);
                    st16(zero, rb_, lane16, off + 1u * MIB);
                    if (DIAG) st16(zero, rb_, lane16, off + 2u * MIB);
                    if (!dry) pending ^= bit[c];
                }
            }
            dry = false;
            if (pending && wt.expired()) dead = true;
        }
        asm volatile("s_waitcnt vmcnt(0)" ::: "memory");
        STAMP(11);
        STAMP_FLUSH();
    }
}

extern "C" void kernel_launch(int rank, int nranks, ncclComm_t comm, hipStream_t stream,
                              void* const* d_in, const int* in_sizes, int n_in,
                              void* d_out, int out_size,
                              const int* in_dtypes, int out_dtype,
                              void* const* d_out_all,
                              void* const* d_ws_all, size_t ws_bytes) {
    if (nranks != NR || (unsigned)rank >= (unsigned)NR || n_in != 2 || ws_bytes < WS_NEED ||
        in_sizes[0] != 2048 * 256 || in_sizes[1] != 256 * 2048 || out_size != 256 * 2048) {
        fprintf(stderr, "kernel_launch: unexpected problem shape\n");
        return;
    }
    static bool attr_set = false;
    if (!attr_set) {
        hipError_t e = hipFuncSetAttribute((const void*)gemm_rs_kernel, hipFuncAttributeMaxDynamicSharedMemorySize, LDS_BYTES);
        if (e != hipSuccess) { fprintf(stderr, "hipFuncSetAttribute: %s\n", hipGetErrorString(e)); return; }
        attr_set = true;
    }
    Params p;
    for (int r = 0; r < NR; ++r) p.ws[r] = (char*)d_ws_all[r];
    p.x = (const char*)d_in[0];
    p.w = (const char*)d_in[1];
    p.out = (char*)d_out;
    p.me = p.ws[rank];
    p.rank = rank;
    gemm_rs_kernel<<<N_WG, 256, LDS_BYTES, stream>>>(p);
}
```

```cpp
#include <hip/hip_runtime.h>
#include <rccl/rccl.h>
#include <cstdio>

typedef unsigned u32;
typedef unsigned long long u64;
typedef u32 u32x4 __attribute__((ext_vector_type(4)));
typedef u32 u32x2 __attribute__((ext_vector_type(2)));
typedef short s16x4 __attribute__((ext_vector_type(4)));
typedef __bf16 bf16x8 __attribute__((ext_vector_type(8)));
typedef __bf16 bf16x2 __attribute__((ext_vector_type(2)));
typedef float f32x2 __attribute__((ext_vector_type(2)));
typedef float f32x16 __attribute__((ext_vector_type(16)));
typedef float f32x4 __attribute__((ext_vector_type(4)));
typedef __amdgpu_buffer_rsrc_t rsrc_t;
typedef __attribute__((address_space(1))) u32 gu32;
#define LDSP(T, p) ((__attribute__((address_space(3))) T*)(p))

constexpr int NR = 8;
constexpr u32 KIB = 1024u, MIB = 1024u * 1024u;
constexpr u32 WS_CTR = 0;
constexpr u32 WS_TMO = 64 * KIB;
constexpr u32 WS_LOG = 128 * KIB;
constexpr u32 WS_OB = 1 * MIB;
constexpr u32 WS_IB = 4 * MIB;
constexpr u32 IB_BYTES = 4 * MIB;
constexpr u32 WS_NEED = WS_IB + 3 * IB_BYTES;
#define NPUSH 64
#define NGEMM 192
constexpr int N_PUSH = NPUSH, N_GEMM = NGEMM, N_WG = 256;
constexpr bool FUSE_PR = N_PUSH + N_GEMM == N_WG;
constexpr int N_RED = FUSE_PR ? N_PUSH : N_WG - N_PUSH - N_GEMM, RED0 = FUSE_PR ? 0 : N_PUSH + N_GEMM;
constexpr u32 N_PAIR = 32 - N_GEMM / 8;
constexpr u32 LDS_BYTES = 128 * KIB;
#define RED_B 4
#define RED_PAR 3
#define OPT_G 4u
#define GEMM_SLEEP "8"
#define RED_SLEEP 4
constexpr u64 DEADLINE_TICKS = 100000000ull;
constexpr int AUX_SYS = 17;

struct Params {
    char* ws[NR];
    const char* x;
    const char* w;
    char* out;
    char* me;
    int rank;
};

__device__ __forceinline__ rsrc_t mk_rsrc(const void* p, u32 bytes) {
    return __builtin_amdgcn_make_buffer_rsrc((void*)p, 0, bytes, 0x00020000);
}
__device__ __forceinline__ void st16(u32x4 v, rsrc_t r, u32 voff, u32 soff) {
    __builtin_amdgcn_raw_buffer_store_b128(v, r, voff + soff, 0, AUX_SYS);
}
__device__ __forceinline__ u32 nz(u32 d) { return d ? d : 0x80000000u; }
__device__ __forceinline__ u32x4 nz4(u32x4 v) { return u32x4{nz(v.x), nz(v.y), nz(v.z), nz(v.w)}; }
__device__ __forceinline__ u32 min4(u32 m, u32x4 v) { return min(min(m, min(v.x, v.y)), min(v.z, v.w)); }
__device__ __forceinline__ void min4_pinned(u32& m, u32x4 v) {
    asm volatile("v_min3_u32 %0, %0, %1, %2\n\tv_min3_u32 %0, %0, %3, %4" : "+v"(m) : "v"(v.x), "v"(v.y), "v"(v.z), "v"(v.w));
}
__device__ __forceinline__ u32 pk_bf16(float a, float b) {
    return __builtin_bit_cast(u32, __builtin_convertvector(f32x2{a, b}, bf16x2));
}
__device__ __forceinline__ float bf_lo(u32 d) { return __builtin_bit_cast(float, d << 16); }
__device__ __forceinline__ float bf_hi(u32 d) { return __builtin_bit_cast(float, d & 0xffff0000u); }
__device__ __forceinline__ float relu(float v) { return v < 0.0f ? 0.0f : v; }
__device__ __forceinline__ u32 add_relu(u32 a, u32 b, u32 c) {
    return pk_bf16(relu(bf_lo(a) + bf_lo(b) + bf_lo(c)), relu(bf_hi(a) + bf_hi(b) + bf_hi(c)));
}

__device__ __forceinline__ bool probe_spin(u32x4 rs, u32 voff, u32 soff, u32& a, u32& b) {
    u32 n;
    asm volatile(
        "s_nop 4\n\t"
        "buffer_load_dword %0, %3, %4, %5 offen sc0 sc1\n\t"
        "s_sleep " GEMM_SLEEP "\n\t"
        "buffer_load_dword %1, %3, %4, %5 offen sc0 sc1\n\t"
        "s_movk_i32 %2, 64\n"
        "1:\n\t"
        "s_waitcnt vmcnt(1)\n\t"
        "v_cmp_ne_u32 vcc, 0, %0\n\t"
        "s_cmp_eq_u64 vcc, exec\n\t"
        "s_cbranch_scc1 2f\n\t"
        "buffer_load_dword %0, %3, %4, %5 offen sc0 sc1\n\t"
        "s_waitcnt vmcnt(1)\n\t"
        "v_cmp_ne_u32 vcc, 0, %1\n\t"
        "s_cmp_eq_u64 vcc, exec\n\t"
        "s_cbranch_scc1 2f\n\t"
        "buffer_load_dword %1, %3, %4, %5 offen sc0 sc1\n\t"
        "s_sub_u32 %2, %2, 1\n\t"
        "s_cmp_lg_u32 %2, 0\n\t"
        "s_cbranch_scc1 1b\n"
        "2:"
        : "=&v"(a), "=&v"(b), "=&s"(n) : "v"(voff), "s"(rs), "s"(soff) : "vcc", "scc", "memory");
    return n != 0u;
}

struct Waiter {
    gu32* tmo; u32 call; u64 t0; u32 spins;
    __device__ __forceinline__ bool expired() {
        asm volatile("" ::: "memory");
        if ((++spins & 63u) != 0) return false;
        return expired_now();
    }
    __device__ __forceinline__ bool expired_now() {
        u32 t = __hip_atomic_load(tmo, __ATOMIC_RELAXED, __HIP_MEMORY_SCOPE_AGENT);
        t = __builtin_amdgcn_readfirstlane(t);
        if (t == call || __builtin_amdgcn_s_memrealtime() - t0 > DEADLINE_TICKS) {
            __hip_atomic_store(tmo, call, __ATOMIC_RELAXED, __HIP_MEMORY_SCOPE_AGENT);
            return true;
        }
        return false;
    }
};

#define PACE16 232
#define LEAD 110
#define ZPACE 7u
#define AUX_BULK 17
#define UMAP 1
#define DIAG 1
#define DIAG_DELAY 32
#define OPACE16 224
#define OLEAD 0
#define GATE 16
#define RGATE 0
#define XSKEW 0
#define WIN "4"
#define AUX_OUT 17
#define AUX_Z 17
#define ENDWAIT 1
#define WARM 0
__device__ __forceinline__ void pace(u32 t0, u32 kib) {
    const u32 due = (u32)LEAD + ((kib * (u32)PACE16) >> 7);
    while ((u32)__builtin_amdgcn_s_memrealtime() - t0 < due) __builtin_amdgcn_s_sleep(1);
}

#define STAMP(i) do {} while (0)
#define STAMPC(i) do {} while (0)
#define STAMP_FLUSH() do {} while (0)

constexpr u32 XS = DIAG ? 7u : 8u;
__device__ __forceinline__ void pace_out(u32 t0, u32 idx) {
    const u32 due = (u32)(LEAD + OLEAD) + (((32u * XS + 256u) * (u32)PACE16) >> 7) + ((idx * (u32)OPACE16) >> 4);
    if (OPACE16) while ((u32)__builtin_amdgcn_s_memrealtime() - t0 < due) __builtin_amdgcn_s_sleep(1);
}

__global__ void __launch_bounds__(256, 1) gemm_rs_kernel(Params p) {
    extern __shared__ __attribute__((aligned(16))) char smem[];
    const u64 t_start = __builtin_amdgcn_s_memrealtime();
    const int b = blockIdx.x;
    const int lane = threadIdx.x & 63;
    const int wave = __builtin_amdgcn_readfirstlane(threadIdx.x >> 6);
    const u32 lane16 = lane * 16;
    constexpr int UPW = 128 / N_PUSH;
    const u32 u = ((u32)b + (u32)wave * N_PUSH) & 127u;
    const u32 xi = u & 1u, kh = (u >> 1) & 1u, xp = (u >> 2) & 3u, xm = u >> 4;
    u32 rank = (u32)p.rank;
    char* me = p.me;
    const char* px = p.x; const char* pw_ = p.w; char* pout = p.out;
    char* d0 = p.ws[b < N_PUSH ? 4u * kh + xi : (u32)b & 7u];
    char* d1 = p.ws[4u * kh + 2u + xi];
    char* d2 = p.ws[4u * kh + 2u * xi];
    char* d3 = p.ws[4u * kh + 2u * xi + 1u];
    asm volatile("" : "+s"(rank), "+s"(me), "+s"(px), "+s"(pw_), "+s"(pout), "+s"(d0), "+s"(d1), "+s"(d2), "+s"(d3));
    rank &= 7u;
    gu32* ctr = (gu32*)(me + WS_CTR + (u32)(b * 4 + wave) * 64u);

    u32 call = __hip_atomic_load(ctr, __ATOMIC_RELAXED, __HIP_MEMORY_SCOPE_AGENT);
    if (!FUSE_PR && b < N_PUSH && wave >= UPW) return;
    if (__builtin_expect(b < N_PUSH && wave < UPW, 1)) {
        const rsrc_t rx = mk_rsrc(px, 2048u * 256u * 2u);
        const rsrc_t rw = mk_rsrc(pw_, 256u * 2048u * 2u);
        u32x4 xv[8], wv[8];
        const u32 xrot = XSKEW ? xm + rank : 0u, xfl = (xrot >> 2) & 1u, xsl = (xp + xrot) & 3u;
        {
            const u32 voff = (u32)(lane & 31) * 512u + (u32)(lane >> 5) * 16u;
            const u32 soff = (256u * xm + 128u * xi) * 512u + kh * 256u + xp * 64u;
#pragma unroll
            for (int f = 0; f < 8; ++f)
                xv[f] = __builtin_amdgcn_raw_buffer_load_b128(rx, voff, soff + (f >> 1) * 32u * 512u + ((f & 1) ^ xfl) * 32u, 0);
        }
        {
            const u32 voff = (u32)(lane >> 2) * 4096u + (u32)(lane & 3) * 16u;
            const u32 soff = (128u * kh + 16u * xm) * 4096u + xi * 2048u + xp * 64u;
#pragma unroll
            for (int f = 0; f < 8; ++f)
                wv[f] = __builtin_amdgcn_raw_buffer_load_b128(rw, voff + f * 256u, soff, 0);
        }
        __builtin_amdgcn_sched_barrier(0);
        const u32 ib = WS_IB + ((__builtin_amdgcn_readfirstlane(call) + 1u) % 3u) * IB_BYTES;
        const u32 tp = (u32)t_start;
        STAMP(1);
        if (!DIAG || xm != rank) {
            const rsrc_t r0 = mk_rsrc(d0 + ib, IB_BYTES), r1 = mk_rsrc(d1 + ib, IB_BYTES);
            const u32 soff = ((xm * 4u + (rank >> 1)) * 64u + (rank & 1u) * 8u + xp * 2u) * KIB;
#pragma unroll
            for (int f = 0; f < 8; ++f) {
                const u32x4 v = nz4(xv[f]);
                if (f == 0) { asm volatile("" :: "v"(v)); STAMP(2); }
                pace(tp, (4u * f + xsl) * XS);
                if (f >= 2) asm volatile("s_waitcnt vmcnt(" WIN ")" ::: "memory");
                st16(v, r0, lane16, soff + (f >> 1) * 16u * KIB + ((f & 1) ^ xfl) * 1024u);
                st16(v, r1, lane16, soff + (f >> 1) * 16u * KIB + ((f & 1) ^ xfl) * 1024u);
                __builtin_amdgcn_sched_barrier(0);
            }
        }
        STAMP(3);
        char* lw = smem + wave * 8 * KIB;
#pragma unroll
        for (int f = 0; f < 8; ++f) *LDSP(u32x4, lw + f * 1024 + lane16) = wv[f];
        const u32 troff = (u32)(8 * (lane >> 5) + ((lane & 15) >> 2)) * 64u + (u32)((lane >> 4) & 1) * 32u + (u32)(lane & 3) * 8u;
        {
            const rsrc_t r0 = mk_rsrc(d2 + ib, IB_BYTES), r1 = mk_rsrc(d3 + ib, IB_BYTES);
            const u32 soff = 2u * MIB + (xp * 64u + rank * 8u + xm) * KIB;
#pragma clang loop unroll(disable)
            for (u32 f = 0; f < 8; ++f) {
                const s16x4 lo = __builtin_amdgcn_ds_read_tr16_b64_v4i16(LDSP(s16x4, lw + f * 1024 + troff));
                const s16x4 hi = __builtin_amdgcn_ds_read_tr16_b64_v4i16(LDSP(s16x4, lw + f * 1024 + troff + 256));
                const u32x2 l2 = __builtin_bit_cast(u32x2, lo), h2 = __builtin_bit_cast(u32x2, hi);
                const u32x4 v = nz4(u32x4{l2.x, l2.y, h2.x, h2.y});
                pace(tp, 32u * XS + (4u * f + xp) * 8u);
                asm volatile("s_waitcnt vmcnt(" WIN ")" ::: "memory");
                st16(v, r0, lane16, soff + f * 256u * KIB);
                st16(v, r1, lane16, soff + f * 256u * KIB);
            }
        }
        STAMP(4);
    }
    call = __builtin_amdgcn_readfirstlane(call) + 1u;
    if (lane == 0) __hip_atomic_store(ctr, call, __ATOMIC_RELAXED, __HIP_MEMORY_SCOPE_AGENT);
    if (!FUSE_PR && b < N_PUSH) { asm volatile("s_waitcnt vmcnt(0)" ::: "memory"); STAMP_FLUSH(); return; }
    Waiter wt{(gu32*)(me + WS_TMO), call, t_start, 0u};
    const u32x4 zero = {0u, 0u, 0u, 0u};
    const u32x4 nanv = {0x7fc07fc0u, 0x7fc07fc0u, 0x7fc07fc0u, 0x7fc07fc0u};

    if (b >= N_PUSH && b < N_PUSH + N_GEMM) {
        const u32 m = (u32)b & 7u;
        const u32 gi = (u32)(b - N_PUSH) >> 3;
        const bool pair = gi < N_PAIR;
        const u32 g = pair ? 2u * gi : gi + N_PAIR;
        char* const ibp = me + WS_IB + (call % 3u) * IB_BYTES;
        if (FUSE_PR && UPW != 2) {
            const rsrc_t rz = mk_rsrc(me + WS_IB + ((call + 2u) % 3u) * IB_BYTES, IB_BYTES);
            for (u32 n = (u32)(b - N_PUSH) * 4u + (u32)wave; n < 4096u; n += N_GEMM * 4u) st16(zero, rz, lane16, n * KIB);
        }
        const u32 canw = (u32)(lane & 15) * 1024u + 1020u;
        const bool hole = DIAG && (u32)wave == (m >> 1), skip_lo = hole && !(m & 1u), skip_hi = hole && (m & 1u);
        const u32 can = skip_lo ? canw | 8192u : skip_hi ? canw & ~8192u : canw;
        const rsrc_t rlo = mk_rsrc(ibp, skip_lo ? 0u : IB_BYTES), rhi = mk_rsrc(ibp, skip_hi ? 0u : IB_BYTES);
#define ALLIN(lo_, hi_) ((__all((lo_) != 0u) || skip_lo) && (__all((hi_) != 0u) || skip_hi))
        const rsrc_t ri = mk_rsrc(ibp, IB_BYTES);
        const rsrc_t ro = mk_rsrc(d0 + WS_OB, 2u * MIB);
        bool dead = false;
#define PROBE1(cn, base) __builtin_amdgcn_raw_buffer_load_b32(ri, cn, (base), AUX_SYS)
        const u32x4 riw = {(u32)(size_t)ibp, (u32)((size_t)ibp >> 32) & 0xffffu, IB_BYTES, 0x00020000u};
        u32 pa = 0, pb = 0;
#define PROBE16(cn, base) do { while (!dead && !probe_spin(riw, cn, (base), pa, pb)) dead = wt.expired_now(); } while (0)
#define KEEP() asm volatile("" :: "v"(pa), "v"(pb))
#define LOADF(base, f_) __builtin_amdgcn_raw_buffer_load_b128((f_) < 8 ? rlo : rhi, lane16 + ((f_) & 3) * 1024u, (base) + ((f_) >> 2) * 4096u, AUX_SYS)
#define SB() __builtin_amdgcn_sched_barrier(0)
#define LOAD16(dst, base) do { _Pragma("unroll") for (int f_ = 0; f_ < 16; ++f_) { dst[f_] = LOADF(base, f_); asm volatile("" ::: "memory"); SB(); } } while (0)
#define FETCH16(dst, base, ncn, next, hit, pin) do { \
            for (;;) { \
                if (!hit) PROBE16(can, base); \
                LOAD16(dst, base); \
                u32 nx_ = PROBE1(ncn, next); \
                u32 lo_ = 0xffffffffu, hi_ = 0xffffffffu; \
                _Pragma("unroll") for (int f_ = 0; f_ < 16; ++f_) { min4_pinned(f_ < 8 ? lo_ : hi_, dst[f_]); if (pin) asm volatile("" : "+a"(dst[f_])); SB(); } \
                KEEP(); \
                asm volatile("" : "+v"(nx_)); \
                hit = __all(nx_ != 0u); \
                if (ALLIN(lo_, hi_) || dead) break; \
                hit = false; \
                if (wt.expired()) dead = true; \
            } \
            } while (0)
        const u32 xbase = (m * 4u + (u32)wave) * 64u * KIB;
        const u32 wbase = 2u * MIB + (g * 64u + (u32)wave * 16u) * KIB;
        u32x4 xf0[16], xf1[16], xf2[16], xf3[16], wf[16], ov[2];
#define LKEY(r) ((((r) >> 1) & 1u) | (((r) & 1u) << 1) | ((r) & 4u))
        char* const pw = smem + (u32)wave * 16u * KIB + (u32)(lane & 31) * 128u;
        const u32 wsw = (u32)(lane >> 5) ^ LKEY((u32)lane);
        char* const pr = smem + (u32)wave * 4u * KIB + (u32)(lane >> 2) * 128u;
        const u32 rsw = (u32)(2 * (lane & 3)) ^ LKEY((u32)lane >> 2);
        const u32 obase = ((rank * 32u + g) * 4u + (u32)wave) * 2048u;
#define MMA(acc, a_, b_) do { acc = __builtin_amdgcn_mfma_f32_32x32x16_bf16(__builtin_bit_cast(bf16x8, a_), __builtin_bit_cast(bf16x8, b_), acc, 0, 0, 0); \
                              asm volatile("" : "+a"(acc)); } while (0)
#define PUT(st, rb, acc, G) *LDSP(f32x4, pw + (st) * 64 * KIB + (rb) * 4096 + ((wsw ^ (2u * (G))) << 4)) = f32x4{acc[4 * (G)], acc[4 * (G) + 1], acc[4 * (G) + 2], acc[4 * (G) + 3]}
#define COMPUTE(st, pref) do { \
            f32x16 acc0 = {}, acc1 = {}, acc2 = {}, acc3 = {}; \
            SB(); \
            _Pragma("unroll") for (int kk = 0; kk < 8; ++kk) { \
                MMA(acc0, wf[kk], xf0[kk]); if (st == 0) min4_pinned(mlo, wf[kk]); SB(); \
                MMA(acc1, wf[kk], xf1[kk]); MMA(acc2, wf[kk], xf2[kk]); MMA(acc3, wf[kk], xf3[kk]); SB(); \
            } \
            _Pragma("unroll") for (int kk = 8; kk < 16; ++kk) { MMA(acc0, wf[kk], xf0[kk]); if (st == 0) min4_pinned(mhi, wf[kk]); SB(); } \
            _Pragma("unroll") for (int kk = 8; kk < 16; ++kk) { MMA(acc1, wf[kk], xf1[kk]); if (kk >= 11 && kk < 15) PUT(st, 0, acc0, kk - 11); \
                if (pref) wf[kk - 8] = LOADF(wbase + 64u * KIB, kk - 8); SB(); } \
            _Pragma("unroll") for (int kk = 8; kk < 16; ++kk) { MMA(acc2, wf[kk], xf2[kk]); if (kk >= 11 && kk < 15) PUT(st, 1, acc1, kk - 11); SB(); } \
            _Pragma("unroll") for (int kk = 8; kk < 16; ++kk) { MMA(acc3, wf[kk], xf3[kk]); if (kk >= 11 && kk < 15) PUT(st, 2, acc2, kk - 11); \
                if (pref) wf[kk] = LOADF(wbase + 64u * KIB, kk); SB(); } \
            PUT(st, 3, acc3, 0); PUT(st, 3, acc3, 1); PUT(st, 3, acc3, 2); PUT(st, 3, acc3, 3); \
            } while (0)
#define POISON(st) do { if (dead) { f32x16 q; for (int i = 0; i < 16; ++i) q[i] = __builtin_nanf(""); \
            for (int rb = 0; rb < 4; ++rb) for (int G = 0; G < 4; ++G) PUT(st, rb, q, G); } } while (0)
#define SUMUP(st, v) do { \
            asm volatile("s_waitcnt lgkmcnt(0)" ::: "memory"); \
            __builtin_amdgcn_s_barrier(); \
            asm volatile("" ::: "memory"); \
            STAMP(5 + 4 * (st)); \
            f32x4 t[16]; \
            _Pragma("unroll") for (int i = 0; i < 16; ++i) \
                t[i] = *LDSP(f32x4, pr + (st) * 64 * KIB + ((i >> 1) & 3) * 16 * KIB + (i >> 3) * 2048 + ((rsw ^ (u32)(i & 1)) << 4)); \
            SB(); \
            _Pragma("unroll") for (int e = 0; e < 2; ++e) { \
                const f32x4 lo = (t[8 * e] + t[8 * e + 2]) + (t[8 * e + 4] + t[8 * e + 6]); \
                const f32x4 hi = (t[8 * e + 1] + t[8 * e + 3]) + (t[8 * e + 5] + t[8 * e + 7]); \
                v[e] = nz4(u32x4{pk_bf16(lo.x, lo.y), pk_bf16(lo.z, lo.w), pk_bf16(hi.x, hi.y), pk_bf16(hi.z, hi.w)}); \
            } } while (0)
        if (DIAG && gi >= N_GEMM / 8 - 16) {
            const u32 dt = (gi - (N_GEMM / 8 - 16)) * 8u + m, dq = dt >> 5, ds = dt & 31u;
            const rsrc_t rx = mk_rsrc(px, 2048u * 256u * 2u);
            const rsrc_t rw = mk_rsrc(pw_, 256u * 2048u * 2u);
            if (DIAG_DELAY) __builtin_amdgcn_s_sleep(DIAG_DELAY);
            u32x4 wv[4];
#pragma unroll
            for (int c = 0; c < 4; ++c)
                wv[c] = __builtin_amdgcn_raw_buffer_load_b128(rw, (u32)(lane >> 2) * 4096u + (u32)(lane & 3) * 16u,
                                                              (64u * (u32)wave + 16u * c) * 4096u + (dq >> 1) * 2048u + ds * 64u, 0);
#pragma unroll
            for (int f = 0; f < 16; ++f)
                wf[f] = __builtin_amdgcn_raw_buffer_load_b128(rx, (u32)(lane & 31) * 512u + (u32)(lane >> 5) * 16u + (f & 3) * 32u,
                                                              (256u * rank + 128u * (dq & 1u) + 32u * (f >> 2)) * 512u + (u32)wave * 128u, 0);
            char* lw = smem + wave * 16 * KIB;
            const u32 troff = (u32)(8 * (lane >> 5) + ((lane & 15) >> 2)) * 64u + (u32)((lane >> 4) & 1) * 32u + (u32)(lane & 3) * 8u;
#pragma unroll
            for (int c = 0; c < 4; ++c) *LDSP(u32x4, lw + c * 1024 + lane16) = wv[c];
#pragma unroll
            for (int c = 0; c < 4; ++c) {
                const u32x2 l2 = __builtin_bit_cast(u32x2, __builtin_amdgcn_ds_read_tr16_b64_v4i16(LDSP(s16x4, lw + c * 1024 + troff)));
                const u32x2 h2 = __builtin_bit_cast(u32x2, __builtin_amdgcn_ds_read_tr16_b64_v4i16(LDSP(s16x4, lw + c * 1024 + troff + 256)));
                wv[c] = u32x4{l2.x, l2.y, h2.x, h2.y};
            }
#pragma unroll
            for (int rb = 0; rb < 4; ++rb) {
                f32x16 acc = {};
#pragma unroll
                for (int c = 0; c < 4; ++c) MMA(acc, wv[c], wf[rb * 4 + c]);
                PUT(1, rb, acc, 0); PUT(1, rb, acc, 1); PUT(1, rb, acc, 2); PUT(1, rb, acc, 3);
            }
            SUMUP(1, ov);
            const rsrc_t rd = mk_rsrc(me + WS_OB + 2u * MIB, 1u * MIB);
            st16(ov[0], rd, lane16, (dt * 4u + (u32)wave) * 2048u);
            st16(ov[1], rd, lane16 + 1024u, (dt * 4u + (u32)wave) * 2048u);
            STAMP(14);
        }
        bool hit = false;
        if (GATE) pace((u32)t_start, 7u * XS + GATE);
        FETCH16(xf0, xbase, can, xbase + 16u * KIB, hit, true);
        STAMP(1);
        if (GATE) pace((u32)t_start, 15u * XS + GATE);
        FETCH16(xf1, xbase + 16u * KIB, can, xbase + 32u * KIB, hit, true);
        if (GATE) pace((u32)t_start, 23u * XS + GATE);
        FETCH16(xf2, xbase + 32u * KIB, can, xbase + 48u * KIB, hit, true);
        if (GATE) pace((u32)t_start, 31u * XS + GATE);
        FETCH16(xf3, xbase + 48u * KIB, canw, wbase, hit, false);
        if (GATE) pace((u32)t_start, 32u * XS + g * 8u + GATE);
        STAMP(2);
        for (;;) {
            if (!hit) PROBE16(canw, wbase);
            LOAD16(wf, wbase);
            STAMP(3); STAMPC(12);
            u32 mlo = 0xffffffffu, mhi = 0xffffffffu;
            if (pair) COMPUTE(0, true); else COMPUTE(0, false);
            KEEP();
            if (ALLIN(mlo, mhi) || dead) break;
            hit = false;
            if (wt.expired()) dead = true;
        }
        POISON(0);
        STAMPC(13); STAMP(4);
        SUMUP(0, ov);
        {
            const u32 j = gi - N_PAIR;
            if (m != rank) pace_out((u32)t_start, pair ? gi : j < 7u ? 8u + j : j < 15u ? 2u * j + 2u : 31u);
        }
        st16(ov[0], ro, lane16, obase);
        st16(ov[1], ro, lane16 + 1024u, obase);
        SB();
        STAMP(6);
        if (pair) {
            u32 mlo = 0xffffffffu, mhi = 0xffffffffu;
#pragma unroll
            for (int f = 0; f < 16; ++f) min4_pinned(f < 8 ? mlo : mhi, wf[f]);
            if (!ALLIN(mlo, mhi)) { hit = false; FETCH16(wf, wbase + 64u * KIB, can, wbase, hit, false); }
            STAMP(7);
            COMPUTE(1, false);
            POISON(1);
            STAMP(8);
            SUMUP(1, ov);
            if (m != rank) pace_out((u32)t_start, 15u + 2u * gi);
            st16(ov[0], ro, lane16, obase + 8192u);
            st16(ov[1], ro, lane16 + 1024u, obase + 8192u);
            STAMP(10);
        }
        asm volatile("s_waitcnt vmcnt(0)" ::: "memory");
        STAMP(11);
        STAMP_FLUSH();
        return;
    }

    {
        const u32 idx = (u32)(b - RED0) * 4u + (u32)wave;
        if (!FUSE_PR) {
            const rsrc_t rz = mk_rsrc(me + WS_IB + ((call + 2u) % 3u) * IB_BYTES, IB_BYTES);
            for (u32 n = idx; n < 4096u; n += N_RED * 4u) st16(zero, rz, lane16, n * KIB);
        }
        if (FUSE_PR && UPW == 2 && wave >= 2) {
            const rsrc_t rz = mk_rsrc(me + WS_IB + ((call + 2u) % 3u) * IB_BYTES, IB_BYTES);
            const u32 z = (u32)b * 2u + (u32)wave - 2u;
#pragma clang loop unroll(disable)
            for (u32 n = 0; n < 32u; ++n) { pace((u32)t_start, n * ZPACE * 2u); st16(zero, rz, lane16, (n * 128u + z) * KIB); }
        }
        STAMP(6);
        constexpr int NPC = 1024 / (N_RED * 4);
        constexpr u32 SG = 32 / NPC;
        static_assert(NPC * N_RED * 4 == 1024 && NPC <= 16, "reducer geometry");
        const u32 quad = (idx >> 3) & 3u, pc = idx & 7u, grp = idx >> 5;
#define PS(j) (SG * (j) + ((grp + (j)) & (SG - 1u)))
#define POFF(j) ((quad * 32u + PS(j)) * 8192u + pc * 1024u)
        const u32 pj = ((u32)lane >> 1) & (NPC - 1u);
        const u32 probe = POFF(pj) + (u32)(lane & 1) * MIB + 1020u;
        const u32 ovoff = (u32)(lane >> 2) * 4096u + (u32)(lane & 3) * 16u + (128u * (quad & 1u) + 16u * pc) * 4096u + (quad >> 1) * 2048u;
        if (RGATE) pace_out((u32)t_start, RGATE);
        bool dead = false, dry = WARM;
        u32 pending = (1u << NPC) - 1u;
        { u32 q = 0; for (int j = 0; j < NPC; ++j) q |= 1u << (2 * j); pending = q; }
        while (pending) {
            const rsrc_t rb_ = mk_rsrc(me + WS_OB, dry ? 0u : 3u * MIB);
            const rsrc_t rout = mk_rsrc(pout, dry ? 0u : 256u * 2048u * 2u);
            u32 act = pending;
            if (__builtin_popcount(pending) > RED_PAR && !dead && !dry) {
                const u32 v = __builtin_amdgcn_raw_buffer_load_b32(rb_, probe, 0u, AUX_SYS);
                u32 got = (u32)__ballot(v != 0u);
                act = got & (got >> 1) & pending;
                if (!act) {
                    if (wt.expired()) dead = true;
                    __builtin_amdgcn_s_sleep(RED_SLEEP);
                    continue;
                }
            }
            u32x4 v0[RED_B], v1[RED_B], v2[RED_B];
            u32 bit[RED_B], pcs[RED_B];
#pragma unroll
            for (int c = 0; c < RED_B; ++c) {
                bit[c] = act & (0u - act); act ^= bit[c];
                const u32 j = bit[c] ? (u32)__builtin_ctz(bit[c]) >> 1 : 0u;
                pcs[c] = PS(j);
                if (bit[c]) {
                    const u32 off = (quad * 32u + pcs[c]) * 8192u + pc * 1024u;
                    v0[c] = __builtin_amdgcn_raw_buffer_load_b128(rb_, lane16, off, AUX_SYS);
                    v1[c] = __builtin_amdgcn_raw_buffer_load_b128(rb_, lane16, off + 1u * MIB, AUX_SYS);
                    if (DIAG) v2[c] = __builtin_amdgcn_raw_buffer_load_b128(rb_, lane16, off + 2u * MIB, AUX_SYS);
                    else v2[c] = u32x4{0x80008000u, 0x80008000u, 0x80008000u, 0x80008000u};
                }
            }
            __builtin_amdgcn_s_waitcnt(0x0F70);
#pragma unroll
            for (int c = 0; c < RED_B; ++c) if (bit[c]) {
                const u32 mn = min4(min4(min4(0xffffffffu, v0[c]), v1[c]), v2[c]);
                if (__all(mn != 0u) || dead || dry) {
                    const u32 off = (quad * 32u + pcs[c]) * 8192u + pc * 1024u;
                    u32x4 r = {add_relu(v0[c].x, v1[c].x, v2[c].x), add_relu(v0[c].y, v1[c].y, v2[c].y), add_relu(v0[c].z, v1[c].z, v2[c].z), add_relu(v0[c].w, v1[c].w, v2[c].w)};
                    if (dead) r = nanv;
                    __builtin_amdgcn_raw_buffer_store_b128(r, rout, ovoff + pcs[c] * 64u, 0, AUX_OUT);
                    __builtin_amdgcn_raw_buffer_store_b128(zero, rb_, lane16 + off, 0, AUX_Z);
                    __builtin_amdgcn_raw_buffer_store_b128(zero, rb_, lane16 + off + 1u * MIB, 0, AUX_Z);
                    if (DIAG) __builtin_amdgcn_raw_buffer_store_b128(zero, rb_, lane16 + off + 2u * MIB, 0, AUX_Z);
                    if (!dry) pending ^= bit[c];
                }
            }
            dry = false;
            if (pending && wt.expired()) dead = true;
        }
        if (ENDWAIT) asm volatile("s_waitcnt vmcnt(0)" ::: "memory");
        STAMP(11);
        STAMP_FLUSH();
    }
}

extern "C" void kernel_launch(int rank, int nranks, ncclComm_t comm, hipStream_t stream,
                              void* const* d_in, const int* in_sizes, int n_in,
                              void* d_out, int out_size,
                              const int* in_dtypes, int out_dtype,
                              void* const* d_out_all,
                              void* const* d_ws_all, size_t ws_bytes) {
    if (nranks != NR || (unsigned)rank >= (unsigned)NR || n_in != 2 || ws_bytes < WS_NEED ||
        in_sizes[0] != 2048 * 256 || in_sizes[1] != 256 * 2048 || out_size != 256 * 2048) {
        fprintf(stderr, "kernel_launch: unexpected problem shape\n");
        return;
    }
    static bool attr_set = false;
    if (!attr_set) {
        hipError_t e = hipFuncSetAttribute((const void*)gemm_rs_kernel, hipFuncAttributeMaxDynamicSharedMemorySize, LDS_BYTES);
        if (e != hipSuccess) { fprintf(stderr, "hipFuncSetAttribute: %s\n", hipGetErrorString(e)); return; }
        attr_set = true;
    }
    Params p;
    for (int r = 0; r < NR; ++r) p.ws[r] = (char*)d_ws_all[r];
    p.x = (const char*)d_in[0];
    p.w = (const char*)d_in[1];
    p.out = (char*)d_out;
    p.me = p.ws[rank];
    p.rank = rank;
    gemm_rs_kernel<<<N_WG, 256, LDS_BYTES, stream>>>(p);
}
```

```cpp
#include <hip/hip_runtime.h>
#include <rccl/rccl.h>
#include <cstdio>

typedef unsigned u32;
typedef unsigned long long u64;
typedef u32 u32x4 __attribute__((ext_vector_type(4)));
typedef u32 u32x2 __attribute__((ext_vector_type(2)));
typedef short s16x4 __attribute__((ext_vector_type(4)));
typedef __bf16 bf16x8 __attribute__((ext_vector_type(8)));
typedef __bf16 bf16x2 __attribute__((ext_vector_type(2)));
typedef float f32x2 __attribute__((ext_vector_type(2)));
typedef float f32x16 __attribute__((ext_vector_type(16)));
typedef float f32x4 __attribute__((ext_vector_type(4)));
typedef __amdgpu_buffer_rsrc_t rsrc_t;
typedef __attribute__((address_space(1))) u32 gu32;
#define LDSP(T, p) ((__attribute__((address_space(3))) T*)(p))

constexpr int NR = 8;
constexpr u32 KIB = 1024u, MIB = 1024u * 1024u;
constexpr u32 WS_CTR = 0;
constexpr u32 WS_TMO = 64 * KIB;
constexpr u32 WS_LOG = 128 * KIB;
constexpr u32 WS_OB = 1 * MIB;
constexpr u32 WS_IB = 4 * MIB;
constexpr u32 IB_BYTES = 4 * MIB;
constexpr u32 WS_NEED = WS_IB + 3 * IB_BYTES;
#define NPUSH 64
#define NGEMM 192
constexpr int N_PUSH = NPUSH, N_GEMM = NGEMM, N_WG = 256;
constexpr bool FUSE_PR = N_PUSH + N_GEMM == N_WG;
constexpr int N_RED = FUSE_PR ? N_PUSH : N_WG - N_PUSH - N_GEMM, RED0 = FUSE_PR ? 0 : N_PUSH + N_GEMM;
constexpr u32 N_PAIR = 32 - N_GEMM / 8;
constexpr u32 LDS_BYTES = 128 * KIB;
#define RED_B 4
#define RED_PAR 3
#define OPT_G 4u
#define GEMM_SLEEP "8"
#define RED_SLEEP 4
constexpr u64 DEADLINE_TICKS = 100000000ull;
constexpr int AUX_SYS = 17;

struct Params {
    char* ws[NR];
    const char* x;
    const char* w;
    char* out;
    char* me;
    int rank;
};

__device__ __forceinline__ rsrc_t mk_rsrc(const void* p, u32 bytes) {
    return __builtin_amdgcn_make_buffer_rsrc((void*)p, 0, bytes, 0x00020000);
}
__device__ __forceinline__ void st16(u32x4 v, rsrc_t r, u32 voff, u32 soff) {
    __builtin_amdgcn_raw_buffer_store_b128(v, r, voff + soff, 0, AUX_SYS);
}
__device__ __forceinline__ u32 nz(u32 d) { return d ? d : 0x80000000u; }
__device__ __forceinline__ u32x4 nz4(u32x4 v) { return u32x4{nz(v.x), nz(v.y), nz(v.z), nz(v.w)}; }
__device__ __forceinline__ u32 min4(u32 m, u32x4 v) { return min(min(m, min(v.x, v.y)), min(v.z, v.w)); }
__device__ __forceinline__ void min4_pinned(u32& m, u32x4 v) {
    asm volatile("v_min3_u32 %0, %0, %1, %2\n\tv_min3_u32 %0, %0, %3, %4" : "+v"(m) : "v"(v.x), "v"(v.y), "v"(v.z), "v"(v.w));
}
__device__ __forceinline__ u32 pk_bf16(float a, float b) {
    return __builtin_bit_cast(u32, __builtin_convertvector(f32x2{a, b}, bf16x2));
}
__device__ __forceinline__ float bf_lo(u32 d) { return __builtin_bit_cast(float, d << 16); }
__device__ __forceinline__ float bf_hi(u32 d) { return __builtin_bit_cast(float, d & 0xffff0000u); }
__device__ __forceinline__ float relu(float v) { return v < 0.0f ? 0.0f : v; }
__device__ __forceinline__ u32 add_relu(u32 a, u32 b, u32 c) {
    return pk_bf16(relu(bf_lo(a) + bf_lo(b) + bf_lo(c)), relu(bf_hi(a) + bf_hi(b) + bf_hi(c)));
}

__device__ __forceinline__ bool probe_spin(u32x4 rs, u32 voff, u32 soff, u32& a, u32& b) {
    u32 n;
    asm volatile(
        "s_nop 4\n\t"
        "buffer_load_dword %0, %3, %4, %5 offen sc0 sc1\n\t"
        "s_sleep " GEMM_SLEEP "\n\t"
        "buffer_load_dword %1, %3, %4, %5 offen sc0 sc1\n\t"
        "s_movk_i32 %2, 64\n"
        "1:\n\t"
        "s_waitcnt vmcnt(1)\n\t"
        "v_cmp_ne_u32 vcc, 0, %0\n\t"
        "s_cmp_eq_u64 vcc, exec\n\t"
        "s_cbranch_scc1 2f\n\t"
        "buffer_load_dword %0, %3, %4, %5 offen sc0 sc1\n\t"
        "s_waitcnt vmcnt(1)\n\t"
        "v_cmp_ne_u32 vcc, 0, %1\n\t"
        "s_cmp_eq_u64 vcc, exec\n\t"
        "s_cbranch_scc1 2f\n\t"
        "buffer_load_dword %1, %3, %4, %5 offen sc0 sc1\n\t"
        "s_sub_u32 %2, %2, 1\n\t"
        "s_cmp_lg_u32 %2, 0\n\t"
        "s_cbranch_scc1 1b\n"
        "2:"
        : "=&v"(a), "=&v"(b), "=&s"(n) : "v"(voff), "s"(rs), "s"(soff) : "vcc", "scc", "memory");
    return n != 0u;
}

struct Waiter {
    gu32* tmo; u32 call; u64 t0; u32 spins;
    __device__ __forceinline__ bool expired() {
        asm volatile("" ::: "memory");
        if ((++spins & 63u) != 0) return false;
        return expired_now();
    }
    __device__ __forceinline__ bool expired_now() {
        u32 t = __hip_atomic_load(tmo, __ATOMIC_RELAXED, __HIP_MEMORY_SCOPE_AGENT);
        t = __builtin_amdgcn_readfirstlane(t);
        if (t == call || __builtin_amdgcn_s_memrealtime() - t0 > DEADLINE_TICKS) {
            __hip_atomic_store(tmo, call, __ATOMIC_RELAXED, __HIP_MEMORY_SCOPE_AGENT);
            return true;
        }
        return false;
    }
};

#define PACE16 224
#define LEAD 80
#define ZPACE 7u
#define AUX_BULK 17
#define UMAP 2
#define DIAG 1
#define DIAG_DELAY 32
#define OPACE16 224
#define OLEAD 0
#define GATE 16
#define RGATE 0
#define XSKEW 1
#define WIN "4"
#define AUX_OUT 17
#define AUX_Z 17
#define ENDWAIT 1
#define NOZERO 0
#define ADAPT 2
#define LATEDBG 0
#define SLOW 0
#define SLOWM 0
#define LAZY 1
#define SDELAY 0u
#define SDEL2 0u
#define TLBW 2
#define GADJ 30
#define MROT 0u
#define NOZ2 0
#define PORD 1
#define WARM 0
__device__ __forceinline__ void pace(u32 t0, u32 kib) {
    const u32 due = (u32)LEAD + ((kib * (u32)PACE16) >> 7);
    while ((u32)__builtin_amdgcn_s_memrealtime() - t0 < due) __builtin_amdgcn_s_sleep(1);
}

#define STAMP(i) do {} while (0)
#define STAMPC(i) do {} while (0)
#define STAMP_FLUSH() do {} while (0)

constexpr u32 XS = DIAG ? 7u : 8u;
__device__ __forceinline__ u32 slot_to_s(u32 idx) {
    return idx < 8u ? 2u * idx : idx < 15u ? idx + 8u : idx == 31u ? 31u : (idx & 1u) ? idx - 14u : 15u + (idx >> 1);
}
__device__ __forceinline__ void pace_out(u32 t0, u32 idx) {
    const u32 due = (u32)(LEAD + OLEAD) + (((32u * XS + 256u) * (u32)PACE16) >> 7) + ((idx * (u32)OPACE16) >> 4);
    if (OPACE16) while ((u32)__builtin_amdgcn_s_memrealtime() - t0 < due) __builtin_amdgcn_s_sleep(1);
}

__global__ void __launch_bounds__(256, 1) gemm_rs_kernel(Params p) {
    extern __shared__ __attribute__((aligned(16))) char smem[];
    const u64 t_start = __builtin_amdgcn_s_memrealtime();
    const int b = blockIdx.x;
    const int lane = threadIdx.x & 63;
    const int wave = __builtin_amdgcn_readfirstlane(threadIdx.x >> 6);
    const u32 lane16 = lane * 16;
    constexpr int UPW = 128 / N_PUSH;
    const u32 xi = (u32)b & 1u, kh = ((u32)b >> 1) & 1u, xp = ((u32)b >> 3) & 3u;
    const u32 xm = ((u32)b & 4u) | (((u32)b >> 5) & 1u) | (((u32)wave & 1u) << 1);
    u32 rank = (u32)p.rank;
    char* me = p.me;
    const char* px = p.x; const char* pw_ = p.w; char* pout = p.out;
    const u32 gm = ((u32)b + MROT * ((u32)b >> 3)) & 7u;
    char* d0 = p.ws[b < N_PUSH ? 4u * kh + xi : gm];
    char* d1 = p.ws[4u * kh + 2u + xi];
    char* d2 = p.ws[4u * kh + 2u * xi];
    char* d3 = p.ws[4u * kh + 2u * xi + 1u];
    asm volatile("" : "+s"(rank), "+s"(me), "+s"(px), "+s"(pw_), "+s"(pout), "+s"(d0), "+s"(d1), "+s"(d2), "+s"(d3));
    rank &= 7u;
    gu32* ctr = (gu32*)(me + WS_CTR + (u32)(b * 4 + wave) * 64u);

    u32 call = __hip_atomic_load(ctr, __ATOMIC_RELAXED, __HIP_MEMORY_SCOPE_AGENT);
    if (!FUSE_PR && b < N_PUSH && wave >= UPW) return;
    if (__builtin_expect(b < N_PUSH && wave < UPW, 1)) {
        const rsrc_t rx = mk_rsrc(px, 2048u * 256u * 2u);
        const rsrc_t rw = mk_rsrc(pw_, 256u * 2048u * 2u);
        u32x4 xv[8], wv[8];
        if (LATEDBG) __builtin_amdgcn_s_sleep(LATEDBG);
        const u32 xvo = (u32)(lane & 31) * 512u + (u32)(lane >> 5) * 16u;
        const u32 xso = (256u * xm + 128u * xi) * 512u + kh * 256u + xp * 64u;
        const u32 xfl = XSKEW ? xm & 1u : 0u, xsl = XSKEW ? xp ^ ((xm >> 1) & 1u) : xp;
#define LDX(f) xv[f] = __builtin_amdgcn_raw_buffer_load_b128(rx, xvo, xso + ((f) >> 1) * 32u * 512u + (((f) & 1) ^ xfl) * 32u, 0)
        const u32 wvo = (u32)(lane >> 2) * 4096u + (u32)(lane & 3) * 16u;
        const u32 wso = (128u * kh + 16u * xm) * 4096u + xi * 2048u + xp * 64u;
#define LDW(f) wv[f] = __builtin_amdgcn_raw_buffer_load_b128(rw, wvo + (f) * 256u, wso, 0)
        LDX(0); LDX(1); LDX(2); LDX(3);
        if (!LAZY) { LDX(4); LDX(5); LDX(6); LDX(7); LDW(0); LDW(1); LDW(2); LDW(3); LDW(4); LDW(5); LDW(6); LDW(7); }
        __builtin_amdgcn_sched_barrier(0);
        const u32 ib = WS_IB + ((__builtin_amdgcn_readfirstlane(call) + 1u) % 3u) * IB_BYTES;
        u32 tp = (u32)t_start;
        {
            asm volatile("" :: "v"(xv[0]));
            const u32 late = (u32)__builtin_amdgcn_s_memrealtime() - tp;
            if (late > (u32)LEAD) tp += late - (u32)LEAD;
        }
        STAMP(2);
        {
            const u32 nb = (!DIAG || xm != rank) ? IB_BYTES : 0u;
            const rsrc_t r0 = mk_rsrc(d0 + ib, nb), r1 = mk_rsrc(d1 + ib, nb);
            const u32 soff = ((xm * 4u + (rank >> 1)) * 64u + (rank & 1u) * 8u + xp * 2u) * KIB;
#pragma unroll
            for (int f = 0; f < 8; ++f) {
                const u32x4 v = nz4(xv[f]);
                pace(tp, (4u * f + xsl) * XS);
                if (f >= 5) asm volatile("s_waitcnt vmcnt(" WIN ")" ::: "memory");
                st16(v, r0, lane16, soff + (f >> 1) * 16u * KIB + ((f & 1) ^ xfl) * 1024u);
                st16(v, r1, lane16, soff + (f >> 1) * 16u * KIB + ((f & 1) ^ xfl) * 1024u);
                __builtin_amdgcn_sched_barrier(0);
                if (LAZY && f == 0) { LDX(4); LDX(5); LDX(6); LDX(7); }
                if (LAZY && f == LAZY) { LDW(0); LDW(1); LDW(2); LDW(3); }
                if (LAZY && f == LAZY + 1) { LDW(4); LDW(5); LDW(6); LDW(7); }
                __builtin_amdgcn_sched_barrier(0);
            }
        }
        STAMP(3);
        char* lw = smem + wave * 8 * KIB;
#pragma unroll
        for (int f = 0; f < 8; ++f) *LDSP(u32x4, lw + f * 1024 + lane16) = wv[f];
        const u32 troff = (u32)(8 * (lane >> 5) + ((lane & 15) >> 2)) * 64u + (u32)((lane >> 4) & 1) * 32u + (u32)(lane & 3) * 8u;
        {
            const rsrc_t r0 = mk_rsrc(d2 + ib, IB_BYTES), r1 = mk_rsrc(d3 + ib, IB_BYTES);
            const u32 soff = 2u * MIB + (xp * 64u + rank * 8u + xm) * KIB;
#pragma clang loop unroll(disable)
            for (u32 f = 0; f < 8; ++f) {
                const s16x4 lo = __builtin_amdgcn_ds_read_tr16_b64_v4i16(LDSP(s16x4, lw + f * 1024 + troff));
                const s16x4 hi = __builtin_amdgcn_ds_read_tr16_b64_v4i16(LDSP(s16x4, lw + f * 1024 + troff + 256));
                const u32x2 l2 = __builtin_bit_cast(u32x2, lo), h2 = __builtin_bit_cast(u32x2, hi);
                const u32x4 v = nz4(u32x4{l2.x, l2.y, h2.x, h2.y});
                pace(tp, 32u * XS + (4u * f + xp) * 8u);
                asm volatile("s_waitcnt vmcnt(" WIN ")" ::: "memory");
                st16(v, r0, lane16, soff + f * 256u * KIB);
                st16(v, r1, lane16, soff + f * 256u * KIB);
            }
        }
        STAMP(4);
    }
    const u32 tlbo = ((lane & 1) ? WS_NEED - 64u * KIB : 512u * KIB) + (u32)b * 128u;
#define TOUCH(ptr, bytes, off) do { u32 t_ = __builtin_amdgcn_raw_buffer_load_b32(mk_rsrc(ptr, bytes), off, 0, AUX_SYS); tl |= t_; } while (0)
    if (TLBW && b < N_PUSH && wave == 2 && lane < 2) {
        u32 tl = 0;
        TOUCH(d0, WS_NEED, tlbo); TOUCH(d1, WS_NEED, tlbo);
        TOUCH(pw_, MIB, (u32)b * 128u); TOUCH(d2, WS_NEED, tlbo); TOUCH(d3, WS_NEED, tlbo); TOUCH(pout, MIB, (u32)b * 128u);
        asm volatile("" :: "v"(tl));
    }
    call = __builtin_amdgcn_readfirstlane(call) + 1u;
    asm volatile("" : "+s"(call));
    const u32 t_call = (u32)__builtin_amdgcn_s_memrealtime();
    if (lane == 0) __hip_atomic_store(ctr, call, __ATOMIC_RELAXED, __HIP_MEMORY_SCOPE_AGENT);
    if (!FUSE_PR && b < N_PUSH) { asm volatile("s_waitcnt vmcnt(0)" ::: "memory"); STAMP_FLUSH(); return; }
    Waiter wt{(gu32*)(me + WS_TMO), call, t_start, 0u};
    const u32x4 zero = {0u, 0u, 0u, 0u};
    const u32x4 nanv = {0x7fc07fc0u, 0x7fc07fc0u, 0x7fc07fc0u, 0x7fc07fc0u};

    if (b >= N_PUSH && b < N_PUSH + N_GEMM) {
        const u32 m = gm;
        const u32 gi = (u32)(b - N_PUSH) >> 3;
        const bool pair = gi < N_PAIR;
        const u32 g = pair ? 2u * gi : gi + N_PAIR;
        char* const ibp = me + WS_IB + (call % 3u) * IB_BYTES;
        if (TLBW && wave == 2 && lane < 2) { u32 tl = 0; TOUCH(d0, WS_NEED, tlbo); asm volatile("" :: "v"(tl)); }
        u32 tg = (u32)t_start;
        if (GADJ >= 0) { const u32 late = t_call - tg + (u32)GADJ; if (late > (u32)LEAD) tg += late - (u32)LEAD; }
        if (FUSE_PR && UPW != 2) {
            const rsrc_t rz = mk_rsrc(me + WS_IB + ((call + 2u) % 3u) * IB_BYTES, IB_BYTES);
            for (u32 n = (u32)(b - N_PUSH) * 4u + (u32)wave; n < 4096u; n += N_GEMM * 4u) st16(zero, rz, lane16, n * KIB);
        }
        const u32 canw = (u32)(lane & 15) * 1024u + 1020u;
        const bool hole = DIAG && (u32)wave == (m >> 1), skip_lo = hole && !(m & 1u), skip_hi = hole && (m & 1u);
        const u32 can = skip_lo ? canw | 8192u : skip_hi ? canw & ~8192u : canw;
        const rsrc_t rlo = mk_rsrc(ibp, skip_lo ? 0u : IB_BYTES), rhi = mk_rsrc(ibp, skip_hi ? 0u : IB_BYTES);
#define ALLIN(lo_, hi_) ((__all((lo_) != 0u) || skip_lo) && (__all((hi_) != 0u) || skip_hi))
        const rsrc_t ri = mk_rsrc(ibp, IB_BYTES);
        const rsrc_t ro = mk_rsrc(d0 + WS_OB, 2u * MIB);
        bool dead = false;
#define PROBE1(cn, base) __builtin_amdgcn_raw_buffer_load_b32(ri, cn, (base), AUX_SYS)
        const u32x4 riw = {(u32)(size_t)ibp, (u32)((size_t)ibp >> 32) & 0xffffu, IB_BYTES, 0x00020000u};
        u32 pa = 0, pb = 0;
#define PROBE16(cn, base) do { while (!dead && !probe_spin(riw, cn, (base), pa, pb)) dead = wt.expired_now(); } while (0)
#define KEEP() asm volatile("" :: "v"(pa), "v"(pb))
#define LOADF(base, f_) __builtin_amdgcn_raw_buffer_load_b128((f_) < 8 ? rlo : rhi, lane16 + ((f_) & 3) * 1024u, (base) + ((f_) >> 2) * 4096u, AUX_SYS)
#define SB() __builtin_amdgcn_sched_barrier(0)
#define LOAD16(dst, base) do { _Pragma("unroll") for (int f_ = 0; f_ < 16; ++f_) { dst[f_] = LOADF(base, f_); asm volatile("" ::: "memory"); SB(); } } while (0)
#define FETCH16(dst, base, ncn, next, hit, pin) do { \
            for (;;) { \
                if (!hit) PROBE16(can, base); \
                LOAD16(dst, base); \
                u32 nx_ = PROBE1(ncn, next); \
                u32 lo_ = 0xffffffffu, hi_ = 0xffffffffu; \
                _Pragma("unroll") for (int f_ = 0; f_ < 16; ++f_) { min4_pinned(f_ < 8 ? lo_ : hi_, dst[f_]); if (pin) asm volatile("" : "+a"(dst[f_])); SB(); } \
                KEEP(); \
                asm volatile("" : "+v"(nx_)); \
                hit = __all(nx_ != 0u); \
                if (ALLIN(lo_, hi_) || dead) break; \
                hit = false; \
                if (wt.expired()) dead = true; \
            } \
            } while (0)
        const u32 xbase = (m * 4u + (u32)wave) * 64u * KIB;
        const u32 wbase = 2u * MIB + (g * 64u + (u32)wave * 16u) * KIB;
        u32x4 xf0[16], xf1[16], xf2[16], xf3[16], wf[16], ov[2];
#define LKEY(r) ((((r) >> 1) & 1u) | (((r) & 1u) << 1) | ((r) & 4u))
        char* const pw = smem + (u32)wave * 16u * KIB + (u32)(lane & 31) * 128u;
        const u32 wsw = (u32)(lane >> 5) ^ LKEY((u32)lane);
        char* const pr = smem + (u32)wave * 4u * KIB + (u32)(lane >> 2) * 128u;
        const u32 rsw = (u32)(2 * (lane & 3)) ^ LKEY((u32)lane >> 2);
        const u32 obase = ((rank * 32u + g) * 4u + (u32)wave) * 2048u;
#define MMA(acc, a_, b_) do { acc = __builtin_amdgcn_mfma_f32_32x32x16_bf16(__builtin_bit_cast(bf16x8, a_), __builtin_bit_cast(bf16x8, b_), acc, 0, 0, 0); \
                              asm volatile("" : "+a"(acc)); } while (0)
#define PUT(st, rb, acc, G) *LDSP(f32x4, pw + (st) * 64 * KIB + (rb) * 4096 + ((wsw ^ (2u * (G))) << 4)) = f32x4{acc[4 * (G)], acc[4 * (G) + 1], acc[4 * (G) + 2], acc[4 * (G) + 3]}
#define COMPUTE(st, pref) do { \
            f32x16 acc0 = {}, acc1 = {}, acc2 = {}, acc3 = {}; \
            SB(); \
            _Pragma("unroll") for (int kk = 0; kk < 8; ++kk) { \
                MMA(acc0, wf[kk], xf0[kk]); if (st == 0) min4_pinned(mlo, wf[kk]); SB(); \
                MMA(acc1, wf[kk], xf1[kk]); MMA(acc2, wf[kk], xf2[kk]); MMA(acc3, wf[kk], xf3[kk]); SB(); \
            } \
            _Pragma("unroll") for (int kk = 8; kk < 16; ++kk) { MMA(acc0, wf[kk], xf0[kk]); if (st == 0) min4_pinned(mhi, wf[kk]); SB(); } \
            _Pragma("unroll") for (int kk = 8; kk < 16; ++kk) { MMA(acc1, wf[kk], xf1[kk]); if (kk >= 11 && kk < 15) PUT(st, 0, acc0, kk - 11); \
                if (pref) wf[kk - 8] = LOADF(wbase + 64u * KIB, kk - 8); SB(); } \
            _Pragma("unroll") for (int kk = 8; kk < 16; ++kk) { MMA(acc2, wf[kk], xf2[kk]); if (kk >= 11 && kk < 15) PUT(st, 1, acc1, kk - 11); SB(); } \
            _Pragma("unroll") for (int kk = 8; kk < 16; ++kk) { MMA(acc3, wf[kk], xf3[kk]); if (kk >= 11 && kk < 15) PUT(st, 2, acc2, kk - 11); \
                if (pref) wf[kk] = LOADF(wbase + 64u * KIB, kk); SB(); } \
            PUT(st, 3, acc3, 0); PUT(st, 3, acc3, 1); PUT(st, 3, acc3, 2); PUT(st, 3, acc3, 3); \
            } while (0)
#define POISON(st) do { if (dead) { f32x16 q; for (int i = 0; i < 16; ++i) q[i] = __builtin_nanf(""); \
            for (int rb = 0; rb < 4; ++rb) for (int G = 0; G < 4; ++G) PUT(st, rb, q, G); } } while (0)
#define SUMUP(st, v) do { \
            asm volatile("s_waitcnt lgkmcnt(0)" ::: "memory"); \
            __builtin_amdgcn_s_barrier(); \
            asm volatile("" ::: "memory"); \
            STAMP(5 + 4 * (st)); \
            f32x4 t[16]; \
            _Pragma("unroll") for (int i = 0; i < 16; ++i) \
                t[i] = *LDSP(f32x4, pr + (st) * 64 * KIB + ((i >> 1) & 3) * 16 * KIB + (i >> 3) * 2048 + ((rsw ^ (u32)(i & 1)) << 4)); \
            SB(); \
            _Pragma("unroll") for (int e = 0; e < 2; ++e) { \
                const f32x4 lo = (t[8 * e] + t[8 * e + 2]) + (t[8 * e + 4] + t[8 * e + 6]); \
                const f32x4 hi = (t[8 * e + 1] + t[8 * e + 3]) + (t[8 * e + 5] + t[8 * e + 7]); \
                v[e] = nz4(u32x4{pk_bf16(lo.x, lo.y), pk_bf16(lo.z, lo.w), pk_bf16(hi.x, hi.y), pk_bf16(hi.z, hi.w)}); \
            } } while (0)
        if (DIAG && gi >= N_GEMM / 8 - 16) {
            const u32 dt = (gi - (N_GEMM / 8 - 16)) * 8u + m, dq = dt >> 5, ds = dt & 31u;
            const rsrc_t rx = mk_rsrc(px, 2048u * 256u * 2u);
            const rsrc_t rw = mk_rsrc(pw_, 256u * 2048u * 2u);
            if (DIAG_DELAY) __builtin_amdgcn_s_sleep(DIAG_DELAY);
            u32x4 wv[4];
#pragma unroll
            for (int c = 0; c < 4; ++c)
                wv[c] = __builtin_amdgcn_raw_buffer_load_b128(rw, (u32)(lane >> 2) * 4096u + (u32)(lane & 3) * 16u,
                                                              (64u * (u32)wave + 16u * c) * 4096u + (dq >> 1) * 2048u + ds * 64u, 0);
#pragma unroll
            for (int f = 0; f < 16; ++f)
                wf[f] = __builtin_amdgcn_raw_buffer_load_b128(rx, (u32)(lane & 31) * 512u + (u32)(lane >> 5) * 16u + (f & 3) * 32u,
                                                              (256u * rank + 128u * (dq & 1u) + 32u * (f >> 2)) * 512u + (u32)wave * 128u, 0);
            char* lw = smem + wave * 16 * KIB;
            const u32 troff = (u32)(8 * (lane >> 5) + ((lane & 15) >> 2)) * 64u + (u32)((lane >> 4) & 1) * 32u + (u32)(lane & 3) * 8u;
#pragma unroll
            for (int c = 0; c < 4; ++c) *LDSP(u32x4, lw + c * 1024 + lane16) = wv[c];
#pragma unroll
            for (int c = 0; c < 4; ++c) {
                const u32x2 l2 = __builtin_bit_cast(u32x2, __builtin_amdgcn_ds_read_tr16_b64_v4i16(LDSP(s16x4, lw + c * 1024 + troff)));
                const u32x2 h2 = __builtin_bit_cast(u32x2, __builtin_amdgcn_ds_read_tr16_b64_v4i16(LDSP(s16x4, lw + c * 1024 + troff + 256)));
                wv[c] = u32x4{l2.x, l2.y, h2.x, h2.y};
            }
#pragma unroll
            for (int rb = 0; rb < 4; ++rb) {
                f32x16 acc = {};
#pragma unroll
                for (int c = 0; c < 4; ++c) MMA(acc, wv[c], wf[rb * 4 + c]);
                PUT(1, rb, acc, 0); PUT(1, rb, acc, 1); PUT(1, rb, acc, 2); PUT(1, rb, acc, 3);
            }
            SUMUP(1, ov);
            const rsrc_t rd = mk_rsrc(me + WS_OB + 2u * MIB, 1u * MIB);
            st16(ov[0], rd, lane16, (dt * 4u + (u32)wave) * 2048u);
            st16(ov[1], rd, lane16 + 1024u, (dt * 4u + (u32)wave) * 2048u);
            STAMP(14);
        }
        bool hit = false;
        if (GATE) pace(tg, 7u * XS + GATE);
        FETCH16(xf0, xbase, can, xbase + 16u * KIB, hit, true);
        STAMP(1);
        if (GATE) pace(tg, 15u * XS + GATE);
        FETCH16(xf1, xbase + 16u * KIB, can, xbase + 32u * KIB, hit, true);
        if (GATE) pace(tg, 23u * XS + GATE);
        FETCH16(xf2, xbase + 32u * KIB, can, xbase + 48u * KIB, hit, true);
        if (GATE) pace(tg, 31u * XS + GATE + (pair ? 0u : SDELAY + SDEL2 * (gi - N_PAIR)));
        FETCH16(xf3, xbase + 48u * KIB, canw, wbase, hit, false);
        if (GATE) pace(tg, 32u * XS + g * 8u + GATE);
        STAMP(2);
        for (;;) {
            if (!hit) PROBE16(canw, wbase);
            LOAD16(wf, wbase);
            STAMP(3); STAMPC(12);
            u32 mlo = 0xffffffffu, mhi = 0xffffffffu;
            if (pair) COMPUTE(0, true); else COMPUTE(0, false);
            KEEP();
            if (ALLIN(mlo, mhi) || dead) break;
            hit = false;
            if (wt.expired()) dead = true;
        }
        POISON(0);
        STAMPC(13); STAMP(4);
        SUMUP(0, ov);
        {
            const u32 j = gi - N_PAIR;
            if (m != rank) pace_out(tg, pair ? gi : j < 7u ? 8u + j : j < 15u ? 2u * j + 2u : 31u);
        }
        st16(ov[0], ro, lane16, obase);
        st16(ov[1], ro, lane16 + 1024u, obase);
        SB();
        STAMP(6);
        if (pair) {
            u32 mlo = 0xffffffffu, mhi = 0xffffffffu;
#pragma unroll
            for (int f = 0; f < 16; ++f) min4_pinned(f < 8 ? mlo : mhi, wf[f]);
            if (!ALLIN(mlo, mhi)) { hit = false; FETCH16(wf, wbase + 64u * KIB, can, wbase, hit, false); }
            STAMP(7);
            COMPUTE(1, false);
            POISON(1);
            STAMP(8);
            SUMUP(1, ov);
            if (m != rank) pace_out(tg, 15u + 2u * gi);
            st16(ov[0], ro, lane16, obase + 8192u);
            st16(ov[1], ro, lane16 + 1024u, obase + 8192u);
            STAMP(10);
        }
        asm volatile("s_waitcnt vmcnt(0)" ::: "memory");
        STAMP(11);
        STAMP_FLUSH();
        return;
    }

    {
        const u32 idx = (u32)(b - RED0) * 4u + (u32)wave;
        if (!FUSE_PR) {
            const rsrc_t rz = mk_rsrc(me + WS_IB + ((call + 2u) % 3u) * IB_BYTES, IB_BYTES);
            for (u32 n = idx; n < 4096u; n += N_RED * 4u) st16(zero, rz, lane16, n * KIB);
        }
        if (!NOZERO && FUSE_PR && UPW == 2 && wave >= 2) {
            const rsrc_t rz = mk_rsrc(me + WS_IB + ((call + 2u) % 3u) * IB_BYTES, IB_BYTES);
            const u32 z = (u32)b * 2u + (u32)wave - 2u;
#pragma clang loop unroll(disable)
            for (u32 n = 0; n < 32u; ++n) { pace((u32)t_start, n * ZPACE * 2u); st16(zero, rz, lane16, (n * 128u + z) * KIB); }
        }
        STAMP(6);
        constexpr int NPC = 1024 / (N_RED * 4);
        constexpr u32 SG = 32 / NPC;
        static_assert(NPC * N_RED * 4 == 1024 && NPC <= 16, "reducer geometry");
        const u32 quad = (idx >> 3) & 3u, pc = idx & 7u, grp = idx >> 5;
#define PS(j) (PORD ? slot_to_s(SG * (j) + grp) : SG * (j) + ((grp + (j)) & (SG - 1u)))
#define POFF(j) ((quad * 32u + PS(j)) * 8192u + pc * 1024u)
        const u32 pj = ((u32)lane >> 1) & (NPC - 1u);
        const u32 probe = POFF(pj) + (u32)(lane & 1) * MIB + 1020u;
        const u32 ovoff = (u32)(lane >> 2) * 4096u + (u32)(lane & 3) * 16u + (128u * (quad & 1u) + 16u * pc) * 4096u + (quad >> 1) * 2048u;
        if (RGATE) pace_out((u32)t_start, RGATE);
        bool dead = false, dry = WARM;
        u32 pending = (1u << NPC) - 1u;
        { u32 q = 0; for (int j = 0; j < NPC; ++j) q |= 1u << (2 * j); pending = q; }
        while (pending) {
            const rsrc_t rb_ = mk_rsrc(me + WS_OB, dry ? 0u : 3u * MIB);
            const rsrc_t rout = mk_rsrc(pout, dry ? 0u : 256u * 2048u * 2u);
            u32 act = pending;
            if (__builtin_popcount(pending) > RED_PAR && !dead && !dry) {
                const u32 v = __builtin_amdgcn_raw_buffer_load_b32(rb_, probe, 0u, AUX_SYS);
                u32 got = (u32)__ballot(v != 0u);
                act = got & (got >> 1) & pending;
                if (!act) {
                    if (wt.expired()) dead = true;
                    __builtin_amdgcn_s_sleep(RED_SLEEP);
                    continue;
                }
            }
            u32x4 v0[RED_B], v1[RED_B], v2[RED_B];
            u32 bit[RED_B], pcs[RED_B];
#pragma unroll
            for (int c = 0; c < RED_B; ++c) {
                bit[c] = act & (0u - act); act ^= bit[c];
                const u32 j = bit[c] ? (u32)__builtin_ctz(bit[c]) >> 1 : 0u;
                pcs[c] = PS(j);
                if (bit[c]) {
                    const u32 off = (quad * 32u + pcs[c]) * 8192u + pc * 1024u;
                    v0[c] = __builtin_amdgcn_raw_buffer_load_b128(rb_, lane16, off, AUX_SYS);
                    v1[c] = __builtin_amdgcn_raw_buffer_load_b128(rb_, lane16, off + 1u * MIB, AUX_SYS);
                    if (DIAG) v2[c] = __builtin_amdgcn_raw_buffer_load_b128(rb_, lane16, off + 2u * MIB, AUX_SYS);
                    else v2[c] = u32x4{0x80008000u, 0x80008000u, 0x80008000u, 0x80008000u};
                }
            }
            __builtin_amdgcn_s_waitcnt(0x0F70);
#pragma unroll
            for (int c = 0; c < RED_B; ++c) if (bit[c]) {
                const u32 mn = min4(min4(min4(0xffffffffu, v0[c]), v1[c]), v2[c]);
                if (__all(mn != 0u) || dead || dry) {
                    const u32 off = (quad * 32u + pcs[c]) * 8192u + pc * 1024u;
                    u32x4 r = {add_relu(v0[c].x, v1[c].x, v2[c].x), add_relu(v0[c].y, v1[c].y, v2[c].y), add_relu(v0[c].z, v1[c].z, v2[c].z), add_relu(v0[c].w, v1[c].w, v2[c].w)};
                    if (dead) r = nanv;
                    __builtin_amdgcn_raw_buffer_store_b128(r, rout, ovoff + pcs[c] * 64u, 0, AUX_OUT);
                    if (!NOZ2) __builtin_amdgcn_raw_buffer_store_b128(zero, rb_, lane16 + off, 0, AUX_Z);
                    if (!NOZ2) __builtin_amdgcn_raw_buffer_store_b128(zero, rb_, lane16 + off + 1u * MIB, 0, AUX_Z);
                    if (DIAG && !NOZ2) __builtin_amdgcn_raw_buffer_store_b128(zero, rb_, lane16 + off + 2u * MIB, 0, AUX_Z);
                    if (!dry) pending ^= bit[c];
                }
            }
            dry = false;
            if (pending && wt.expired()) dead = true;
        }
        if (ENDWAIT) asm volatile("s_waitcnt vmcnt(0)" ::: "memory");
        STAMP(11);
        STAMP_FLUSH();
    }
}

extern "C" void kernel_launch(int rank, int nranks, ncclComm_t comm, hipStream_t stream,
                              void* const* d_in, const int* in_sizes, int n_in,
                              void* d_out, int out_size,
                              const int* in_dtypes, int out_dtype,
                              void* const* d_out_all,
                              void* const* d_ws_all, size_t ws_bytes) {
    if (nranks != NR || (unsigned)rank >= (unsigned)NR || n_in != 2 || ws_bytes < WS_NEED ||
        in_sizes[0] != 2048 * 256 || in_sizes[1] != 256 * 2048 || out_size != 256 * 2048) {
        fprintf(stderr, "kernel_launch: unexpected problem shape\n");
        return;
    }
    static bool attr_set = false;
    if (!attr_set) {
        hipError_t e = hipFuncSetAttribute((const void*)gemm_rs_kernel, hipFuncAttributeMaxDynamicSharedMemorySize, LDS_BYTES);
        if (e != hipSuccess) { fprintf(stderr, "hipFuncSetAttribute: %s\n", hipGetErrorString(e)); return; }
        attr_set = true;
    }
    Params p;
    for (int r = 0; r < NR; ++r) p.ws[r] = (char*)d_ws_all[r];
    p.x = (const char*)d_in[0];
    p.w = (const char*)d_in[1];
    p.out = (char*)d_out;
    p.me = p.ws[rank];
    p.rank = rank;
    gemm_rs_kernel<<<N_WG, 256, LDS_BYTES, stream>>>(p);
}
```

```cpp
#include <hip/hip_runtime.h>
#include <rccl/rccl.h>
#include <cstdio>

typedef unsigned u32;
typedef unsigned long long u64;
typedef u32 u32x4 __attribute__((ext_vector_type(4)));
typedef u32 u32x2 __attribute__((ext_vector_type(2)));
typedef short s16x4 __attribute__((ext_vector_type(4)));
typedef __bf16 bf16x8 __attribute__((ext_vector_type(8)));
typedef __bf16 bf16x2 __attribute__((ext_vector_type(2)));
typedef float f32x2 __attribute__((ext_vector_type(2)));
typedef float f32x4 __attribute__((ext_vector_type(4)));
typedef float f32x16 __attribute__((ext_vector_type(16)));
typedef __amdgpu_buffer_rsrc_t rsrc_t;
typedef __attribute__((address_space(1))) u32 gu32;
#define LDSP(T, p) ((__attribute__((address_space(3))) T*)(p))
#define SB() __builtin_amdgcn_sched_barrier(0)

constexpr int NR = 8;
constexpr u32 KIB = 1024u, MIB = 1024u * 1024u;
constexpr u32 WS_CTR = 0;
constexpr u32 WS_TMO = 64 * KIB;
constexpr u32 WS_OB = 1 * MIB;
constexpr u32 WS_IB = 4 * MIB;
constexpr u32 IB_BYTES = 4 * MIB;
constexpr u32 WS_NEED = WS_IB + 3 * IB_BYTES;
constexpr u32 IN_BYTES = 2048u * 256u * 2u;
constexpr int N_PUSH = 64, N_WG = 256;
constexpr u32 N_PAIR = 8;
constexpr u32 LDS_BYTES = 128 * KIB;
constexpr u64 DEADLINE_TICKS = 100000000ull;
constexpr int AUX_SYS = 17;
constexpr u32 PACE16 = 224;
constexpr u32 LEAD = 80;
constexpr u32 XS = 7;

struct Params {
    char* ws[NR];
    const char* x;
    const char* w;
    char* out;
    char* me;
    int rank;
};

__device__ __forceinline__ rsrc_t mk_rsrc(const void* p, u32 bytes) {
    return __builtin_amdgcn_make_buffer_rsrc((void*)p, 0, bytes, 0x00020000);
}
__device__ __forceinline__ void st16(u32x4 v, rsrc_t r, u32 voff, u32 soff) {
    __builtin_amdgcn_raw_buffer_store_b128(v, r, voff + soff, 0, AUX_SYS);
}
__device__ __forceinline__ u32 nz(u32 d) { return d ? d : 0x80000000u; }
__device__ __forceinline__ u32x4 nz4(u32x4 v) { return u32x4{nz(v.x), nz(v.y), nz(v.z), nz(v.w)}; }
__device__ __forceinline__ u32 min4(u32 m, u32x4 v) { return min(min(m, min(v.x, v.y)), min(v.z, v.w)); }
__device__ __forceinline__ void min4_pinned(u32& m, u32x4 v) {
    asm volatile("v_min3_u32 %0, %0, %1, %2\n\tv_min3_u32 %0, %0, %3, %4" : "+v"(m) : "v"(v.x), "v"(v.y), "v"(v.z), "v"(v.w));
}
__device__ __forceinline__ u32 pk_bf16(float a, float b) {
    return __builtin_bit_cast(u32, __builtin_convertvector(f32x2{a, b}, bf16x2));
}
__device__ __forceinline__ float bf_lo(u32 d) { return __builtin_bit_cast(float, d << 16); }
__device__ __forceinline__ float bf_hi(u32 d) { return __builtin_bit_cast(float, d & 0xffff0000u); }
__device__ __forceinline__ float relu(float v) { return v < 0.0f ? 0.0f : v; }
__device__ __forceinline__ u32 add_relu(u32 a, u32 b, u32 c) {
    return pk_bf16(relu(bf_lo(a) + bf_lo(b) + bf_lo(c)), relu(bf_hi(a) + bf_hi(b) + bf_hi(c)));
}
__device__ __forceinline__ u32x4 tr_frag(char* blk, u32 troff) {
    const u32x2 lo = __builtin_bit_cast(u32x2, __builtin_amdgcn_ds_read_tr16_b64_v4i16(LDSP(s16x4, blk + troff)));
    const u32x2 hi = __builtin_bit_cast(u32x2, __builtin_amdgcn_ds_read_tr16_b64_v4i16(LDSP(s16x4, blk + troff + 256)));
    return u32x4{lo.x, lo.y, hi.x, hi.y};
}

__device__ __forceinline__ bool probe_spin(u32x4 rs, u32 voff, u32 soff, u32& a, u32& b) {
    u32 n;
    asm volatile(
        "s_nop 4\n\t"
        "buffer_load_dword %0, %3, %4, %5 offen sc0 sc1\n\t"
        "s_sleep 8\n\t"
        "buffer_load_dword %1, %3, %4, %5 offen sc0 sc1\n\t"
        "s_movk_i32 %2, 64\n"
        "1:\n\t"
        "s_waitcnt vmcnt(1)\n\t"
        "v_cmp_ne_u32 vcc, 0, %0\n\t"
        "s_cmp_eq_u64 vcc, exec\n\t"
        "s_cbranch_scc1 2f\n\t"
        "buffer_load_dword %0, %3, %4, %5 offen sc0 sc1\n\t"
        "s_waitcnt vmcnt(1)\n\t"
        "v_cmp_ne_u32 vcc, 0, %1\n\t"
        "s_cmp_eq_u64 vcc, exec\n\t"
        "s_cbranch_scc1 2f\n\t"
        "buffer_load_dword %1, %3, %4, %5 offen sc0 sc1\n\t"
        "s_sub_u32 %2, %2, 1\n\t"
        "s_cmp_lg_u32 %2, 0\n\t"
        "s_cbranch_scc1 1b\n"
        "2:"
        : "+v"(a), "+v"(b), "=&s"(n) : "v"(voff), "s"(rs), "s"(soff) : "vcc", "scc", "memory");
    return n != 0u;
}

struct Waiter {
    gu32* tmo; u32 call; u64 t0; u32 spins;
    __device__ __forceinline__ bool expired() {
        asm volatile("" ::: "memory");
        if ((++spins & 63u) != 0) return false;
        return expired_now();
    }
    __device__ __forceinline__ bool expired_now() {
        u32 t = __hip_atomic_load(tmo, __ATOMIC_RELAXED, __HIP_MEMORY_SCOPE_AGENT);
        t = __builtin_amdgcn_readfirstlane(t);
        if (t == call || __builtin_amdgcn_s_memrealtime() - t0 > DEADLINE_TICKS) {
            __hip_atomic_store(tmo, call, __ATOMIC_RELAXED, __HIP_MEMORY_SCOPE_AGENT);
            return true;
        }
        return false;
    }
};

__device__ __forceinline__ void pace(u32 t0, u32 kib) {
    const u32 due = LEAD + ((kib * PACE16) >> 7);
    while ((u32)__builtin_amdgcn_s_memrealtime() - t0 < due) __builtin_amdgcn_s_sleep(1);
}
__device__ __forceinline__ void pace_out(u32 t0, u32 slot) { pace(t0, 32u * XS + 256u + 8u * slot); }
__device__ __forceinline__ u32 slot_to_s(u32 slot) {
    return slot < 8u ? 2u * slot : slot < 15u ? slot + 8u : slot == 31u ? 31u : (slot & 1u) ? slot - 14u : 15u + (slot >> 1);
}

__global__ void __launch_bounds__(256, 1) gemm_rs_kernel(Params p) {
    extern __shared__ __attribute__((aligned(16))) char smem[];
    const u64 t_start = __builtin_amdgcn_s_memrealtime();
    const int b = blockIdx.x;
    const int lane = threadIdx.x & 63;
    const int wave = __builtin_amdgcn_readfirstlane(threadIdx.x >> 6);
    u32 lane16 = lane * 16;
    const u32 xi = (u32)b & 1u, kh = ((u32)b >> 1) & 1u, xp = ((u32)b >> 3) & 3u;
    const u32 xm = ((u32)b & 4u) | (((u32)b >> 5) & 1u) | (((u32)wave & 1u) << 1);
    u32 rank = (u32)p.rank;
    char* me = p.me;
    const char* px = p.x; const char* pw_ = p.w; char* pout = p.out;
    char* d0 = p.ws[b < N_PUSH ? 4u * kh + xi : (u32)b & 7u];
    char* d1 = p.ws[4u * kh + 2u + xi];
    char* d2 = p.ws[4u * kh + 2u * xi];
    char* d3 = p.ws[4u * kh + 2u * xi + 1u];
    u32 xvo = (u32)(lane & 31) * 512u + (u32)(lane >> 5) * 16u;
    const u32 xfl = xm & 1u;
    u32 xso0 = (256u * xm + 128u * xi) * 512u + kh * 256u + xp * 64u + xfl * 32u, xso1 = xso0 ^ 32u;
    u32 not_push = ((u32)b >> 6) | ((u32)wave >> 1);
    u32 ctro = WS_CTR + (u32)(b * 4 + wave) * 64u;
    u32 wvo = (u32)(lane >> 2) * 4096u + (u32)(lane & 3) * 16u;
    u32 troff = (u32)(8 * (lane >> 5) + ((lane & 15) >> 2)) * 64u + (u32)((lane >> 4) & 1) * 32u + (u32)(lane & 3) * 8u;
    asm volatile("" : "+v"(xvo), "+v"(ctro), "+v"(wvo), "+v"(troff), "+v"(lane16));
    asm volatile("" : "+s"(xso0), "+s"(xso1), "+s"(not_push));
    const bool is_push = not_push == 0u;
    asm volatile("" : "+s"(rank), "+s"(me), "+s"(px), "+s"(pw_), "+s"(pout), "+s"(d0), "+s"(d1), "+s"(d2), "+s"(d3));
    gu32* ctr = (gu32*)(me + ctro);
    u32 call = __hip_atomic_load(ctr, __ATOMIC_RELAXED, __HIP_MEMORY_SCOPE_AGENT);
    SB();
    const rsrc_t rx = mk_rsrc(px, IN_BYTES);
    rank &= 7u;
    const rsrc_t rw = mk_rsrc(pw_, IN_BYTES);

    if (__builtin_expect(is_push, 1)) {
        u32x4 xv[8], wv[8];
#define LDX(f) xv[f] = __builtin_amdgcn_raw_buffer_load_b128(rx, xvo, (((f) & 1) ? xso1 : xso0) + ((f) >> 1) * 32u * 512u, 0)
        LDX(0); LDX(1); LDX(2); LDX(3);
        SB();
        const u32 xsl = xp ^ ((xm >> 1) & 1u);
        const u32 wso = (128u * kh + 16u * xm) * 4096u + xi * 2048u + xp * 64u;
#define LDW(f) wv[f] = __builtin_amdgcn_raw_buffer_load_b128(rw, wvo + (f) * 256u, wso, 0)
        const u32 ib = WS_IB + ((__builtin_amdgcn_readfirstlane(call) + 1u) % 3u) * IB_BYTES;
        u32 tp = (u32)t_start;
        {
            u32 nb = xm != rank ? IB_BYTES : 0u;
            char* e0 = d0 + ib; char* e1 = d1 + ib;
            asm volatile("" : "+s"(nb), "+s"(e0), "+s"(e1));
            const rsrc_t r0 = mk_rsrc(e0, nb), r1 = mk_rsrc(e1, nb);
            const u32 soff = ((xm * 4u + (rank >> 1)) * 64u + (rank & 1u) * 8u + xp * 2u) * KIB;
#pragma unroll
            for (int f = 0; f < 8; ++f) {
                u32x4 v = nz4(xv[f]);
                u32 vo = lane16 + soff + (f >> 1) * 16u * KIB + ((f & 1) ^ xfl) * 1024u;
                asm volatile("" : "+v"(v), "+v"(vo));
                if (f == 0) {
                    const u32 late = (u32)__builtin_amdgcn_s_memrealtime() - tp;
                    if (late > LEAD) tp += late - LEAD;
                }
                pace(tp, (4u * f + xsl) * XS);
                if (f >= 5) asm volatile("s_waitcnt vmcnt(4)" ::: "memory");
                st16(v, r0, vo, 0u);
                st16(v, r1, vo, 0u);
                SB();
                if (f == 0) { LDX(4); LDX(5); LDX(6); LDX(7); }
                if (f == 1) { LDW(0); LDW(1); LDW(2); LDW(3); }
                if (f == 2) { LDW(4); LDW(5); LDW(6); LDW(7); }
                SB();
            }
        }
        char* lw = smem + wave * 8 * KIB;
#pragma unroll
        for (int f = 0; f < 8; ++f) *LDSP(u32x4, lw + f * 1024 + lane16) = wv[f];
        {
            const rsrc_t r0 = mk_rsrc(d2 + ib, IB_BYTES), r1 = mk_rsrc(d3 + ib, IB_BYTES);
            const u32 soff = 2u * MIB + (xp * 64u + rank * 8u + xm) * KIB;
#pragma clang loop unroll(disable)
            for (u32 f = 0; f < 8; ++f) {
                const u32x4 v = nz4(tr_frag(lw + f * 1024, troff));
                pace(tp, 32u * XS + (4u * f + xp) * 8u);
                asm volatile("s_waitcnt vmcnt(4)" ::: "memory");
                st16(v, r0, lane16, soff + f * 256u * KIB);
                st16(v, r1, lane16, soff + f * 256u * KIB);
            }
        }
    }
    const u32 tlbo = ((lane & 1) ? WS_NEED - 64u * KIB : 512u * KIB) + (u32)b * 128u;
#define TOUCH(ptr, bytes, off) tl |= __builtin_amdgcn_raw_buffer_load_b32(mk_rsrc(ptr, bytes), off, 0, AUX_SYS)
    if (b < N_PUSH && wave == 2 && lane < 2) {
        u32 tl = 0;
        TOUCH(d0, WS_NEED, tlbo); TOUCH(d1, WS_NEED, tlbo); TOUCH(pw_, IN_BYTES, (u32)b * 128u);
        TOUCH(d2, WS_NEED, tlbo); TOUCH(d3, WS_NEED, tlbo); TOUCH(pout, IN_BYTES, (u32)b * 128u);
        asm volatile("" :: "v"(tl));
    }
    call = __builtin_amdgcn_readfirstlane(call) + 1u;
    asm volatile("" : "+s"(call));
    const u32 t_call = (u32)__builtin_amdgcn_s_memrealtime();
    if (lane == 0) __hip_atomic_store(ctr, call, __ATOMIC_RELAXED, __HIP_MEMORY_SCOPE_AGENT);
    Waiter wt{(gu32*)(me + WS_TMO), call, t_start, 0u};
    const u32x4 zero = {0u, 0u, 0u, 0u};
    bool dead = false;

    if (b >= N_PUSH) {
        const u32 m = (u32)b & 7u;
        const u32 gi = (u32)(b - N_PUSH) >> 3;
        const bool pair = gi < N_PAIR;
        const u32 g = pair ? 2u * gi : gi + N_PAIR;
        char* const ibp = me + WS_IB + (call % 3u) * IB_BYTES;
        if (wave == 2 && lane < 2) { u32 tl = 0; TOUCH(d0, WS_NEED, tlbo); asm volatile("" :: "v"(tl)); }
        u32 tg = (u32)t_start;
        { const u32 late = t_call - tg + 30u; if (late > LEAD) tg += late - LEAD; }
        const bool hole = (u32)wave == (m >> 1), skip_lo = hole && !(m & 1u), skip_hi = hole && (m & 1u);
        const u32 canw = (u32)(lane & 15) * 1024u + 1020u;
        const u32 canx = skip_lo ? canw | 8192u : skip_hi ? canw & ~8192u : canw;
        const rsrc_t rlo = mk_rsrc(ibp, skip_lo ? 0u : IB_BYTES), rhi = mk_rsrc(ibp, skip_hi ? 0u : IB_BYTES);
        const rsrc_t ri = mk_rsrc(ibp, IB_BYTES);
        const rsrc_t ro = mk_rsrc(d0 + WS_OB, 2u * MIB);
        const u32x4 riw = {(u32)(size_t)ibp, (u32)((size_t)ibp >> 32) & 0xffffu, IB_BYTES, 0x00020000u};
        u32 pa = 0, pb = 0;
#define ALLIN(lo_, hi_) ((__all((lo_) != 0u) || skip_lo) && (__all((hi_) != 0u) || skip_hi))
#define PROBE1(cn, base) __builtin_amdgcn_raw_buffer_load_b32(ri, cn, (base), AUX_SYS)
#define PROBE16(cn, base) do { while (!dead && !probe_spin(riw, cn, (base), pa, pb)) dead = wt.expired_now(); } while (0)
#define KEEP() asm volatile("" :: "v"(pa), "v"(pb))
#define LOADF(base, f_) __builtin_amdgcn_raw_buffer_load_b128((f_) < 8 ? rlo : rhi, lane16 + ((f_) & 3) * 1024u, (base) + ((f_) >> 2) * 4096u, AUX_SYS)
#define LOAD16(dst, base) do { _Pragma("unroll") for (int f_ = 0; f_ < 16; ++f_) { dst[f_] = LOADF(base, f_); asm volatile("" ::: "memory"); SB(); } } while (0)
#define FETCH16(dst, base, ncn, next, pin) do { \
            for (;;) { \
                if (!hit) PROBE16(canx, base); \
                LOAD16(dst, base); \
                u32 nx_ = PROBE1(ncn, next); \
                u32 lo_ = 0xffffffffu, hi_ = 0xffffffffu; \
                _Pragma("unroll") for (int f_ = 0; f_ < 16; ++f_) { min4_pinned(f_ < 8 ? lo_ : hi_, dst[f_]); if (pin) asm volatile("" : "+a"(dst[f_])); SB(); } \
                KEEP(); \
                asm volatile("" : "+v"(nx_)); \
                hit = __all(nx_ != 0u); \
                if (ALLIN(lo_, hi_) || dead) break; \
                hit = false; \
                if (wt.expired()) dead = true; \
            } } while (0)
        const u32 xbase = (m * 4u + (u32)wave) * 64u * KIB;
        const u32 wbase = 2u * MIB + (g * 64u + (u32)wave * 16u) * KIB;
        u32x4 xf0[16], xf1[16], xf2[16], xf3[16], wf[16], ov[2];
#define LKEY(r) ((((r) >> 1) & 1u) | (((r) & 1u) << 1) | ((r) & 4u))
        char* const pw = smem + (u32)wave * 16u * KIB + (u32)(lane & 31) * 128u;
        const u32 wsw = (u32)(lane >> 5) ^ LKEY((u32)lane);
        char* const pr = smem + (u32)wave * 4u * KIB + (u32)(lane >> 2) * 128u;
        const u32 rsw = (u32)(2 * (lane & 3)) ^ LKEY((u32)lane >> 2);
        const u32 obase = ((rank * 32u + g) * 4u + (u32)wave) * 2048u;
#define MMA(acc, a_, b_) do { acc = __builtin_amdgcn_mfma_f32_32x32x16_bf16(__builtin_bit_cast(bf16x8, a_), __builtin_bit_cast(bf16x8, b_), acc, 0, 0, 0); \
                              asm volatile("" : "+a"(acc)); } while (0)
#define PUT(st, rb, acc, G) *LDSP(f32x4, pw + (st) * 64 * KIB + (rb) * 4096 + ((wsw ^ (2u * (G))) << 4)) = f32x4{acc[4 * (G)], acc[4 * (G) + 1], acc[4 * (G) + 2], acc[4 * (G) + 3]}
#define COMPUTE(st, pref) do { \
            f32x16 acc0 = {}, acc1 = {}, acc2 = {}, acc3 = {}; \
            SB(); \
            _Pragma("unroll") for (int kk = 0; kk < 8; ++kk) { \
                MMA(acc0, wf[kk], xf0[kk]); if (st == 0) min4_pinned(mlo, wf[kk]); SB(); \
                MMA(acc1, wf[kk], xf1[kk]); MMA(acc2, wf[kk], xf2[kk]); MMA(acc3, wf[kk], xf3[kk]); SB(); \
            } \
            _Pragma("unroll") for (int kk = 8; kk < 16; ++kk) { MMA(acc0, wf[kk], xf0[kk]); if (st == 0) min4_pinned(mhi, wf[kk]); SB(); } \
            _Pragma("unroll") for (int kk = 8; kk < 16; ++kk) { MMA(acc1, wf[kk], xf1[kk]); if (kk >= 11 && kk < 15) PUT(st, 0, acc0, kk - 11); \
                if (pref) wf[kk - 8] = LOADF(wbase + 64u * KIB, kk - 8); SB(); } \
            _Pragma("unroll") for (int kk = 8; kk < 16; ++kk) { MMA(acc2, wf[kk], xf2[kk]); if (kk >= 11 && kk < 15) PUT(st, 1, acc1, kk - 11); SB(); } \
            _Pragma("unroll") for (int kk = 8; kk < 16; ++kk) { MMA(acc3, wf[kk], xf3[kk]); if (kk >= 11 && kk < 15) PUT(st, 2, acc2, kk - 11); \
                if (pref) wf[kk] = LOADF(wbase + 64u * KIB, kk); SB(); } \
            PUT(st, 3, acc3, 0); PUT(st, 3, acc3, 1); PUT(st, 3, acc3, 2); PUT(st, 3, acc3, 3); \
            } while (0)
#define POISON(st) do { if (dead) { f32x16 q; for (int i = 0; i < 16; ++i) q[i] = __builtin_nanf(""); \
            for (int rb = 0; rb < 4; ++rb) for (int G = 0; G < 4; ++G) PUT(st, rb, q, G); } } while (0)
#define SUMUP(st, v) do { \
            asm volatile("s_waitcnt lgkmcnt(0)" ::: "memory"); \
            __builtin_amdgcn_s_barrier(); \
            asm volatile("" ::: "memory"); \
            f32x4 t[16]; \
            _Pragma("unroll") for (int i = 0; i < 16; ++i) \
                t[i] = *LDSP(f32x4, pr + (st) * 64 * KIB + ((i >> 1) & 3) * 16 * KIB + (i >> 3) * 2048 + ((rsw ^ (u32)(i & 1)) << 4)); \
            SB(); \
            _Pragma("unroll") for (int e = 0; e < 2; ++e) { \
                const f32x4 lo = (t[8 * e] + t[8 * e + 2]) + (t[8 * e + 4] + t[8 * e + 6]); \
                const f32x4 hi = (t[8 * e + 1] + t[8 * e + 3]) + (t[8 * e + 5] + t[8 * e + 7]); \
                v[e] = nz4(u32x4{pk_bf16(lo.x, lo.y), pk_bf16(lo.z, lo.w), pk_bf16(hi.x, hi.y), pk_bf16(hi.z, hi.w)}); \
            } } while (0)
        if (!pair) {
            const u32 dt = (gi - N_PAIR) * 8u + m, dq = dt >> 5, ds = dt & 31u;
            __builtin_amdgcn_s_sleep(32);
            u32x4 wv[4];
#pragma unroll
            for (int c = 0; c < 4; ++c)
                wv[c] = __builtin_amdgcn_raw_buffer_load_b128(rw, wvo, (64u * (u32)wave + 16u * c) * 4096u + (dq >> 1) * 2048u + ds * 64u, 0);
#pragma unroll
            for (int f = 0; f < 16; ++f)
                wf[f] = __builtin_amdgcn_raw_buffer_load_b128(rx, xvo + (f & 3) * 32u,
                                                              (256u * rank + 128u * (dq & 1u) + 32u * (f >> 2)) * 512u + (u32)wave * 128u, 0);
            char* lw = smem + wave * 16 * KIB;
#pragma unroll
            for (int c = 0; c < 4; ++c) *LDSP(u32x4, lw + c * 1024 + lane16) = wv[c];
#pragma unroll
            for (int c = 0; c < 4; ++c) wv[c] = tr_frag(lw + c * 1024, troff);
#pragma unroll
            for (int rb = 0; rb < 4; ++rb) {
                f32x16 acc = {};
#pragma unroll
                for (int c = 0; c < 4; ++c) MMA(acc, wv[c], wf[rb * 4 + c]);
                PUT(1, rb, acc, 0); PUT(1, rb, acc, 1); PUT(1, rb, acc, 2); PUT(1, rb, acc, 3);
            }
            SUMUP(1, ov);
            const rsrc_t rd = mk_rsrc(me + WS_OB + 2u * MIB, 1u * MIB);
            st16(ov[0], rd, lane16, (dt * 4u + (u32)wave) * 2048u);
            st16(ov[1], rd, lane16 + 1024u, (dt * 4u + (u32)wave) * 2048u);
        }
        bool hit = false;
        pace(tg, 7u * XS + 16u);
        FETCH16(xf0, xbase, canx, xbase + 16u * KIB, true);
        pace(tg, 15u * XS + 16u);
        FETCH16(xf1, xbase + 16u * KIB, canx, xbase + 32u * KIB, true);
        pace(tg, 23u * XS + 16u);
        FETCH16(xf2, xbase + 32u * KIB, canx, xbase + 48u * KIB, true);
        pace(tg, 31u * XS + 16u);
        FETCH16(xf3, xbase + 48u * KIB, canw, wbase, false);
        pace(tg, 32u * XS + g * 8u + 16u);
        for (;;) {
            if (!hit) PROBE16(canw, wbase);
            LOAD16(wf, wbase);
            u32 mlo = 0xffffffffu, mhi = 0xffffffffu;
            if (pair) COMPUTE(0, true); else COMPUTE(0, false);
            KEEP();
            if (ALLIN(mlo, mhi) || dead) break;
            hit = false;
            if (wt.expired()) dead = true;
        }
        POISON(0);
        SUMUP(0, ov);
        {
            const u32 j = gi - N_PAIR;
            if (m != rank) pace_out(tg, pair ? gi : j < 7u ? 8u + j : j < 15u ? 2u * j + 2u : 31u);
        }
        st16(ov[0], ro, lane16, obase);
        st16(ov[1], ro, lane16 + 1024u, obase);
        SB();
        if (pair) {
            u32 mlo = 0xffffffffu, mhi = 0xffffffffu;
#pragma unroll
            for (int f = 0; f < 16; ++f) min4_pinned(f < 8 ? mlo : mhi, wf[f]);
            if (!ALLIN(mlo, mhi)) { hit = false; FETCH16(wf, wbase + 64u * KIB, canx, wbase, false); }
            COMPUTE(1, false);
            POISON(1);
            SUMUP(1, ov);
            if (m != rank) pace_out(tg, 15u + 2u * gi);
            st16(ov[0], ro, lane16, obase + 8192u);
            st16(ov[1], ro, lane16 + 1024u, obase + 8192u);
        }
        asm volatile("s_waitcnt vmcnt(0)" ::: "memory");
        return;
    }

    if (wave >= 2) {
        const rsrc_t rz = mk_rsrc(me + WS_IB + ((call + 2u) % 3u) * IB_BYTES, IB_BYTES);
        const u32 z = (u32)b * 2u + (u32)wave - 2u;
#pragma clang loop unroll(disable)
        for (u32 n = 0; n < 32u; ++n) { pace((u32)t_start, n * 14u); st16(zero, rz, lane16, (n * 128u + z) * KIB); }
    }
    const u32 idx = (u32)b * 4u + (u32)wave;
    const u32 quad = (idx >> 3) & 3u, pc = idx & 7u, grp = idx >> 5;
    const u32 pbase = quad * 32u * 8192u + pc * 1024u;
    const u32 probe = pbase + slot_to_s(8u * (((u32)lane >> 1) & 3u) + grp) * 8192u + (u32)(lane & 1) * MIB + 1020u;
    const u32 ovoff = wvo + (128u * (quad & 1u) + 16u * pc) * 4096u + (quad >> 1) * 2048u;
    const rsrc_t rp = mk_rsrc(me + WS_OB, 3u * MIB);
    const rsrc_t rout = mk_rsrc(pout, IN_BYTES);
    u32 pending = 0x55u;
    while (pending) {
        u32 act = pending;
        if (pending == 0x55u && !dead) {
            const u32 v = __builtin_amdgcn_raw_buffer_load_b32(rp, probe, 0u, AUX_SYS);
            const u32 got = (u32)__ballot(v != 0u);
            act = got & (got >> 1) & pending;
            if (!act) {
                if (wt.expired()) dead = true;
                __builtin_amdgcn_s_sleep(4);
                continue;
            }
        }
        u32x4 v0[4], v1[4], v2[4];
        u32 bit[4], pcs[4];
#pragma unroll
        for (int c = 0; c < 4; ++c) {
            bit[c] = act & (0u - act); act ^= bit[c];
            const u32 j = bit[c] ? (u32)__builtin_ctz(bit[c]) >> 1 : 0u;
            pcs[c] = slot_to_s(8u * j + grp);
            if (bit[c]) {
                const u32 off = pbase + pcs[c] * 8192u;
                v0[c] = __builtin_amdgcn_raw_buffer_load_b128(rp, lane16, off, AUX_SYS);
                v1[c] = __builtin_amdgcn_raw_buffer_load_b128(rp, lane16, off + 1u * MIB, AUX_SYS);
                v2[c] = __builtin_amdgcn_raw_buffer_load_b128(rp, lane16, off + 2u * MIB, AUX_SYS);
            }
        }
        __builtin_amdgcn_s_waitcnt(0x0F70);
#pragma unroll
        for (int c = 0; c < 4; ++c) if (bit[c]) {
            const u32 mn = min4(min4(min4(0xffffffffu, v0[c]), v1[c]), v2[c]);
            if (__all(mn != 0u) || dead) {
                const u32 off = pbase + pcs[c] * 8192u;
                u32x4 r = {add_relu(v0[c].x, v1[c].x, v2[c].x), add_relu(v0[c].y, v1[c].y, v2[c].y),
                           add_relu(v0[c].z, v1[c].z, v2[c].z), add_relu(v0[c].w, v1[c].w, v2[c].w)};
                if (dead) r = u32x4{0x7fc07fc0u, 0x7fc07fc0u, 0x7fc07fc0u, 0x7fc07fc0u};
                st16(r, rout, ovoff, pcs[c] * 64u);
                st16(zero, rp, lane16, off);
                st16(zero, rp, lane16, off + 1u * MIB);
                st16(zero, rp, lane16, off + 2u * MIB);
                pending ^= bit[c];
            }
        }
        if (pending && wt.expired()) dead = true;
    }
    asm volatile("s_waitcnt vmcnt(0)" ::: "memory");
}

extern "C" void kernel_launch(int rank, int nranks, ncclComm_t comm, hipStream_t stream,
                              void* const* d_in, const int* in_sizes, int n_in,
                              void* d_out, int out_size,
                              const int* in_dtypes, int out_dtype,
                              void* const* d_out_all,
                              void* const* d_ws_all, size_t ws_bytes) {
    if (nranks != NR || (unsigned)rank >= (unsigned)NR || n_in != 2 || ws_bytes < WS_NEED ||
        in_sizes[0] != 2048 * 256 || in_sizes[1] != 256 * 2048 || out_size != 256 * 2048) {
        fprintf(stderr, "kernel_launch: unexpected problem shape\n");
        return;
    }
    static bool attr_set = false;
    if (!attr_set) {
        hipError_t e = hipFuncSetAttribute((const void*)gemm_rs_kernel, hipFuncAttributeMaxDynamicSharedMemorySize, LDS_BYTES);
        if (e != hipSuccess) { fprintf(stderr, "hipFuncSetAttribute: %s\n", hipGetErrorString(e)); return; }
        attr_set = true;
    }
    Params p;
    for (int r = 0; r < NR; ++r) p.ws[r] = (char*)d_ws_all[r];
    p.x = (const char*)d_in[0];
    p.w = (const char*)d_in[1];
    p.out = (char*)d_out;
    p.me = p.ws[rank];
    p.rank = rank;
    gemm_rs_kernel<<<N_WG, 256, LDS_BYTES, stream>>>(p);
}
```
